```python
import numpy as np

kernel_name = "gpu_gemm_ar_m4096_k4096_n16384_e4m3e4m3_dqrelu"


ABSMAX_RTOL = 0.03

COMPUTE_DTYPE = "float32"
EPILOGUE = "dequant_relu"
IN_QUANT = None
OUT_QUANT = None
IN_FP8 = "e4m3e4m3"
IN_DTYPES = {"x": "e4m3", "w_mat": "e4m3",
             "scale_x": "float32", "scale_w": "float32"}
WORLD_SIZE = 8

def _snap_fp8(x, mant_bits, max_abs, min_normal):
    a = np.abs(x.astype(np.float64))
    _m, _e = np.frexp(a)
    q = np.where(a >= min_normal, np.ldexp(1.0, _e - 1 - mant_bits),
                 min_normal * 2.0 ** -mant_bits)
    snapped = np.minimum(np.round(a / q) * q, max_abs)
    return (np.sign(x) * snapped).astype(np.float32)

def setup_inputs():

    x = _snap_fp8(np.random.randn(4096, 4096).astype(np.float32), 3, 448.0, 2.0 ** -6)
    w_mat = _snap_fp8(np.random.randn(4096, 16384).astype(np.float32), 3, 448.0, 2.0 ** -6)
    scale_x = (0.02 * (1.0 + np.random.rand(1))).astype(np.float32)
    scale_w = (0.02 * (1.0 + np.random.rand(1))).astype(np.float32)
    return {"x": x, "w_mat": w_mat, "scale_x": scale_x, "scale_w": scale_w}

def reference(x, w_mat, scale_x, scale_w):
    acc = (x @ w_mat).astype(np.float32)
    y = (acc * (scale_x * scale_w)).astype(np.float32)
    return np.maximum(y, 0.0).astype(np.float32)
```

```cpp
#include <hip/hip_runtime.h>
#include <rccl/rccl.h>
#include <cstdio>

#define HIP_CHECK(expr) do { hipError_t _e = (expr); if (_e != hipSuccess) { \
    fprintf(stderr, "HIP error %s at %s:%d\n", hipGetErrorString(_e), __FILE__, __LINE__); \
    return; } } while (0)

namespace {
#define LAS __attribute__((address_space(3)))
typedef int i32x4 __attribute__((ext_vector_type(4)));
typedef int i32x8 __attribute__((ext_vector_type(8)));
typedef float f32x4 __attribute__((ext_vector_type(4)));
typedef unsigned u32x4 __attribute__((ext_vector_type(4)));
typedef unsigned char u8;

constexpr int M = 4096, N = 16384, K = 4096, NR = 8, KS = K / NR;
constexpr int HTB = 16384, BLK = 32768, STAGE_BYTES = 8 * HTB, LDS_BYTES = STAGE_BYTES + 64;
constexpr int NT = 32, NI = 16, NJ = 64;
#define NCOMM 16
constexpr int C_WG = NCOMM;
constexpr int GRID = 256;
constexpr int L_WG = (GRID - C_WG) / 8;
constexpr int TPX = 128;
constexpr size_t WS_CALL = 0, WS_DONE = 128, WS_TMO = 256;
constexpr size_t WS_FLAGS = 65536;
constexpr int NXC = 16, NWC = 128, NCHUNK = NXC + NWC;
constexpr size_t WS_DATA = 1 << 20;
constexpr size_t A_BYTES = (size_t)M * K, B_BYTES = (size_t)K * N, PAR_BYTES = A_BYTES + B_BYTES;
constexpr size_t WS_NEED = WS_DATA + 2 * PAR_BYTES;
constexpr unsigned long long DEADLINE = 50000000ull;
constexpr int XUNITS = 32, WUNITS = 16;

struct Params { const u8* x; const u8* w; const float* sx; const float* sw; float* out; u8* ws[NR]; int rank; };

__device__ __forceinline__ int lds_byte(int r, int c) { const int st = (r >> 4) * 2 + (c >> 6), ob = (r & 15) * 64 + (c & 63); return st * 1024 + (ob ^ (((ob >> 9) & 1) << 5)); }
__device__ __forceinline__ int xchunk(int kk, int quarter) { return kk * 4 + quarter; }
__device__ __forceinline__ int wchunk(int pair, int kk) { return NXC + pair * 4 + kk; }

#define WAIT_V(n) asm volatile("s_waitcnt vmcnt(" #n ")" ::: "memory")
#define WAIT_L(n) asm volatile("s_waitcnt lgkmcnt(" #n ")" ::: "memory")
#define BAR __builtin_amdgcn_s_barrier()
#define SCHED __builtin_amdgcn_sched_barrier(0)

constexpr int U_HEAD = 4 * 256, U_TOTAL = U_HEAD + 24 * 64;
struct UnitDesc { int isx, kk, a, b, chunk; };
__device__ __forceinline__ UnitDesc unit_desc(int u) {
    UnitDesc d;
    if (u < U_HEAD) { const int kk = u >> 8, r = u & 255; d.kk = kk;
        if (r < 128) { d.isx = 1; d.a = r >> 3; d.b = r & 7; d.chunk = xchunk(kk, r >> 5); }
        else { const int q = r - 128; d.isx = 0; d.a = q >> 3; d.b = q & 7; d.chunk = wchunk(q >> 4, kk); }
    } else { const int v = u - U_HEAD; const int pair = 8 + (v >> 6), kk = (v >> 4) & 3, q = v & 15; d.isx = 0; d.kk = kk; d.a = pair * 2 + (q >> 3); d.b = q & 7; d.chunk = wchunk(pair, kk); }
    return d;
}

__device__ __forceinline__ void comm_role(const Params& p, unsigned call, int cw  ) {
    const int lane = threadIdx.x & 63;
    const int rank = p.rank;
    const size_t par = WS_DATA + (size_t)(call & 1) * PAR_BYTES;
    __amdgpu_buffer_rsrc_t rs[NR];
#pragma unroll
    for (int d = 0; d < NR; ++d) rs[d] = __builtin_amdgcn_make_buffer_rsrc(p.ws[(rank + 1 + d) & 7] + par, 0, (int)PAR_BYTES, 0x00020000);
    unsigned* myflag = (unsigned*)(p.ws[(rank + 1 + (lane & 7)) & 7] + WS_FLAGS) + rank;
    constexpr int NW = C_WG * 8;
    const int s = lane >> 2, cqp = lane & 3;
    const int cq = cqp ^ ((s >> 3) << 1);
    const int quad = 4 * (s & 3) + (s >> 2);
    u32x4 cur[4], nxt[4];
    unsigned ld[16];
    auto issue_loads = [&](const UnitDesc& d) {
        if (d.isx) {
#pragma unroll
            for (int i = 0; i < 4; ++i) { const int st = (d.b & 3) * 4 + i;
                const int R = (d.b >> 2) * 128 + (st >> 1) * 16 + s, Cb = (st & 1) * 64 + cq * 16;
                const u32x4 v = *(const u32x4*)(p.x + (size_t)(d.a * 256 + R) * KS + d.kk * 128 + Cb);
                ld[4 * i] = v[0]; ld[4 * i + 1] = v[1]; ld[4 * i + 2] = v[2]; ld[4 * i + 3] = v[3]; }
        } else {
            const int wc = d.b >> 1, kh = d.b & 1;
            const u8* src = p.w + (size_t)(d.kk * 128 + kh * 64 + cq * 16) * N + d.a * 256 + wc * 64 + quad * 4;
#pragma unroll
            for (int i = 0; i < 16; ++i) ld[i] = *(const unsigned*)(src + (size_t)i * N);
        }
    };
    auto transform = [&](const UnitDesc& d) {
        if (d.isx) {
#pragma unroll
            for (int i = 0; i < 4; ++i) cur[i] = (u32x4){ld[4 * i], ld[4 * i + 1], ld[4 * i + 2], ld[4 * i + 3]};
        } else {
#pragma unroll
            for (int t = 0; t < 4; ++t) {
                const unsigned d0 = ld[4 * t], d1 = ld[4 * t + 1], d2 = ld[4 * t + 2], d3 = ld[4 * t + 3];
                const unsigned a = __builtin_amdgcn_perm(d1, d0, 0x05010400u), b = __builtin_amdgcn_perm(d1, d0, 0x07030602u);
                const unsigned c = __builtin_amdgcn_perm(d3, d2, 0x05010400u), e = __builtin_amdgcn_perm(d3, d2, 0x07030602u);
                cur[0][t] = __builtin_amdgcn_perm(c, a, 0x05040100u); cur[1][t] = __builtin_amdgcn_perm(c, a, 0x07060302u);
                cur[2][t] = __builtin_amdgcn_perm(e, b, 0x05040100u); cur[3][t] = __builtin_amdgcn_perm(e, b, 0x07060302u);
            }
        }
    };
    int u = cw;
    if (u >= U_TOTAL) return;
    UnitDesc d = unit_desc(u);
    issue_loads(d);
    int prev_chunk = -1;
    for (;;) {
        transform(d);
        unsigned off[4];
        if (d.isx) {
#pragma unroll
            for (int i = 0; i < 4; ++i) off[i] = (unsigned)(((d.kk * 8 + rank) * NI + d.a) * BLK + d.b * 4096 + i * 1024 + lane * 16);
        } else {
            const int wc = d.b >> 1, kh = d.b & 1;
#pragma unroll
            for (int jj = 0; jj < 4; ++jj) off[jj] = (unsigned)(A_BYTES + (size_t)((d.kk * 8 + rank) * NJ + d.a) * BLK + (jj >> 1) * HTB + ((wc * 2 + (jj & 1)) * 2 + kh) * 1024 + lane * 16);
        }
        const int un = u + NW; const bool more = un < U_TOTAL;
        UnitDesc dn = unit_desc(more ? un : u);
        const int this_chunk = d.chunk;
        if (more) issue_loads(dn);
        SCHED;
#pragma unroll
        for (int q = 0; q < NR; ++q)
#pragma unroll
            for (int i = 0; i < 4; ++i) __builtin_amdgcn_raw_buffer_store_b128(cur[i], rs[q], off[i], 0, 17);
        if (prev_chunk >= 0) {
            if (!more) WAIT_V(32); else if (dn.isx) WAIT_V(36); else WAIT_V(48);
            if (lane < 8) __hip_atomic_fetch_add(myflag + prev_chunk * 16, 1u, __ATOMIC_RELAXED, __HIP_MEMORY_SCOPE_SYSTEM);
        }
        prev_chunk = this_chunk;
        if (!more) break;
        u = un; d = dn;
    }
    WAIT_V(0);
    if (lane < 8) __hip_atomic_fetch_add(myflag + prev_chunk * 16, 1u, __ATOMIC_RELAXED, __HIP_MEMORY_SCOPE_SYSTEM);
}

__global__ __launch_bounds__(512, 2) void fused_k(Params p) {
    extern __shared__ __attribute__((aligned(16))) u8 lds_raw[];
    LAS u8* lds = (LAS u8*)lds_raw;
    const int tid = threadIdx.x, wid = __builtin_amdgcn_readfirstlane(tid >> 6), lane = tid & 63;
    const int bx = blockIdx.x, rank = p.rank;
    u8* myws = p.ws[rank];
    const unsigned call = (unsigned)__builtin_amdgcn_readfirstlane(__hip_atomic_load((unsigned*)(myws + WS_CALL), __ATOMIC_RELAXED, __HIP_MEMORY_SCOPE_AGENT)) + 1u;
    unsigned* tmo = (unsigned*)(myws + WS_TMO);

    if (bx < C_WG) {
        comm_role(p, call, bx * 8 + wid);
    } else {
        const int wr = wid >> 2, wc = wid & 3, fr = lane & 15, fq = lane >> 4;
        const unsigned voff = (unsigned)tid * 16u;
        const unsigned ldsw = (unsigned)wid * 1024u;
        const int aoff = lds_byte(wr * 64 + fr, fq * 16), boff = lds_byte(wc * 32 + fr, fq * 16);
        volatile LAS int* lknown = (volatile LAS int*)(lds + STAGE_BYTES);
#define SA(b, h) (((b) * 2 + (h)) * HTB)
#define SB(b, h) ((4 + (b) * 2 + (h)) * HTB)
#define STAGE(bufoff, gbase) do { _Pragma("unroll") for (int _i = 0; _i < 2; ++_i) \
        __builtin_amdgcn_global_load_lds((const unsigned*)((const char*)(gbase) + voff + _i * 8192), (LAS unsigned*)(lds + (bufoff) + ldsw + _i * 8192), 16, 0, 0); } while (0)
#define LDF(off) __builtin_shufflevector(*(const LAS i32x4*)(lds + (off)), *(const LAS i32x4*)(lds + (off) + 1024), 0, 1, 2, 3, 4, 5, 6, 7)
#define LDA(dst, b, h) do { _Pragma("unroll") for (int m = 0; m < 4; ++m) dst[m] = LDF(SA(b, h) + aoff + m * 2048); } while (0)
#define LDB(dst, b, h) do { _Pragma("unroll") for (int n = 0; n < 2; ++n) dst[n] = LDF(SB(b, h) + boff + n * 2048); } while (0)
#define MMA(ai, bj, At, Bt) asm volatile("s_setprio 1\n\t" \
        "v_mfma_f32_16x16x128_f8f6f4 %0, %12, %8, %0\n\t"  "v_mfma_f32_16x16x128_f8f6f4 %1, %13, %8, %1\n\t" \
        "v_mfma_f32_16x16x128_f8f6f4 %2, %12, %9, %2\n\t"  "v_mfma_f32_16x16x128_f8f6f4 %3, %13, %9, %3\n\t" \
        "v_mfma_f32_16x16x128_f8f6f4 %4, %12, %10, %4\n\t" "v_mfma_f32_16x16x128_f8f6f4 %5, %13, %10, %5\n\t" \
        "v_mfma_f32_16x16x128_f8f6f4 %6, %12, %11, %6\n\t" "v_mfma_f32_16x16x128_f8f6f4 %7, %13, %11, %7\n\t" \
        "s_setprio 0" \
        : "+v"(acc[ai][bj][0][0]), "+v"(acc[ai][bj][0][1]), "+v"(acc[ai][bj][1][0]), "+v"(acc[ai][bj][1][1]), "+v"(acc[ai][bj][2][0]), "+v"(acc[ai][bj][2][1]), "+v"(acc[ai][bj][3][0]), "+v"(acc[ai][bj][3][1]) \
        : "v"(At[0]), "v"(At[1]), "v"(At[2]), "v"(At[3]), "v"(Bt[0]), "v"(Bt[1]))
#define ZERO_ACC() do { _Pragma("unroll") for (int a = 0; a < 2; ++a) _Pragma("unroll") for (int b = 0; b < 2; ++b) { _Pragma("unroll") for (int m = 0; m < 4; ++m) _Pragma("unroll") for (int n = 0; n < 2; ++n) acc[a][b][m][n] = (f32x4){0.f, 0.f, 0.f, 0.f}; \
        asm volatile("s_nop 1" : "+v"(acc[a][b][0][0]), "+v"(acc[a][b][0][1]), "+v"(acc[a][b][1][0]), "+v"(acc[a][b][1][1]), "+v"(acc[a][b][2][0]), "+v"(acc[a][b][2][1]), "+v"(acc[a][b][3][0]), "+v"(acc[a][b][3][1])); } } while (0)

        const int xcd = bx & 7, loc = (bx >> 3) - C_WG / 8;
        const int rq = xcd & 3, cpar = xcd >> 2;
        const size_t par = WS_DATA + (size_t)(call & 1) * PAR_BYTES;
        const char* Abase = (const char*)myws + par;
        const char* Bbase = Abase + A_BYTES;
        const unsigned* flags = (const unsigned*)(myws + WS_FLAGS);
        bool bad = false;
        int known = 0;
        auto ready = [&](int pair, int need, bool first_tile) {
            if (known >= need) return;
            if (wid == 0) {
                const int kk = (lane >> 3) & 3, src = lane & 7;
                const bool isx = lane >= 32;
                const unsigned* f = flags + (isx ? xchunk(kk, rq) : wchunk(pair, kk)) * 16 + src;
                const unsigned want = call * (isx ? XUNITS : WUNITS);
                const unsigned long long t0 = __builtin_amdgcn_s_memrealtime();
                int kn;
                for (;;) {
                    unsigned v = want;
                    if (!isx || first_tile) v = __hip_atomic_load(f, __ATOMIC_RELAXED, __HIP_MEMORY_SCOPE_SYSTEM);
                    const unsigned long long okm = __ballot(v >= want);
                    const unsigned long long both = okm & (okm >> 32);
                    kn = 0;
                    if ((both & 0xffull) == 0xffull) { kn = 1; if (((both >> 8) & 0xffull) == 0xffull) { kn = 2; if (((both >> 16) & 0xffull) == 0xffull) { kn = 3; if (((both >> 24) & 0xffull) == 0xffull) kn = 4; } } }
                    if (kn >= need) break;
                    if ((unsigned)__builtin_amdgcn_readfirstlane(__hip_atomic_load(tmo, __ATOMIC_RELAXED, __HIP_MEMORY_SCOPE_AGENT)) == call || __builtin_amdgcn_s_memrealtime() - t0 > DEADLINE) {
                        if (lane == 0) __hip_atomic_store(tmo, call, __ATOMIC_RELAXED, __HIP_MEMORY_SCOPE_AGENT);
                        kn = 4; break; }
                    __builtin_amdgcn_s_sleep(2);
                }
                __builtin_amdgcn_fence(__ATOMIC_ACQUIRE, "");
                if (lane == 0) *lknown = kn;
                asm volatile("s_waitcnt vmcnt(0) lgkmcnt(0)" ::: "memory");
            }
            asm volatile("" ::: "memory"); BAR; asm volatile("" ::: "memory");
            known = __builtin_amdgcn_readfirstlane(*lknown);
            asm volatile("s_waitcnt lgkmcnt(0)" ::: "memory");
        };
        auto kt = [&](int n) { return (n & ~7) | ((rank + n) & 7); };
        if (loc < TPX) {
        int u = loc;
        int ti = rq * 4 + (u & 3), pair = u >> 2, tj = pair * 2 + cpar;
        f32x4 acc[2][2][4][2];
        ZERO_ACC();
        i32x8 At[4], B0[2], B1[2];
        const float sc = p.sx[0] * p.sw[0];
        const char* cA = Abase + (size_t)ti * BLK;
        const char* cB = Bbase + (size_t)tj * BLK;
#define APTR(base, n) ((base) + (size_t)kt(n) * (NI * BLK))
#define BPTR(base, n) ((base) + (size_t)kt(n) * (NJ * BLK))
        ready(pair, 1, true);
        {
            const char* a0 = APTR(cA, 0); const char* b0 = BPTR(cB, 0); const char* a1 = APTR(cA, 1); const char* b1 = BPTR(cB, 1);
            STAGE(SB(0, 0), b0); STAGE(SA(0, 0), a0); STAGE(SB(0, 1), b0 + HTB); STAGE(SA(0, 1), a0 + HTB);
            if (wr == 1) BAR;
            WAIT_V(4); BAR;
            STAGE(SB(1, 0), b1); STAGE(SA(1, 0), a1); STAGE(SB(1, 1), b1 + HTB);
            WAIT_V(6); BAR;
        }
        bool first = true;
        for (;;) {
            const int un = u + L_WG; const bool has_next = un < TPX;
            const int nti = rq * 4 + (un & 3), npair = un >> 2, ntj = npair * 2 + cpar;
            const char* nA = has_next ? Abase + (size_t)nti * BLK : cA;
            const char* nB = has_next ? Bbase + (size_t)ntj * BLK : cB;
            for (int t = 0; t < NT; t += 2) {
                const bool last = (t == NT - 2);
                if (last) { if (has_next) { known = 0; ready(npair, 1, false); } }
                else if (((t + 2) & 7) == 0) ready(pair, ((t + 2) >> 3) + 1, first);
                const char* a1 = APTR(cA, t + 1);
                const char* a2 = last ? APTR(nA, 0) : APTR(cA, t + 2); const char* b2 = last ? BPTR(nB, 0) : BPTR(cB, t + 2);
                const char* a3 = last ? APTR(nA, 1) : APTR(cA, t + 3); const char* b3 = last ? BPTR(nB, 1) : BPTR(cB, t + 3);
                LDB(B0, 0, 0); SCHED; LDA(At, 0, 0); STAGE(SA(1, 1), a1 + HTB);
                WAIT_L(8); BAR; WAIT_L(0); MMA(0, 0, At, B0); BAR; SCHED;
                LDB(B1, 0, 1); STAGE(SB(0, 0), b2);
                BAR; WAIT_L(0); MMA(0, 1, At, B1); BAR;
                LDA(At, 0, 1); STAGE(SA(0, 0), a2);
                BAR; WAIT_L(0); MMA(1, 0, At, B0); BAR; SCHED;
                STAGE(SB(0, 1), b2 + HTB);
                WAIT_V(6); BAR; MMA(1, 1, At, B1); BAR;
                LDB(B0, 1, 0); SCHED; LDA(At, 1, 0); STAGE(SA(0, 1), a2 + HTB);
                WAIT_L(8); BAR; WAIT_L(0); MMA(0, 0, At, B0); BAR; SCHED;
                LDB(B1, 1, 1); STAGE(SB(1, 0), b3);
                BAR; WAIT_L(0); MMA(0, 1, At, B1); BAR;
                LDA(At, 1, 1); STAGE(SA(1, 0), a3);
                BAR; WAIT_L(0); MMA(1, 0, At, B0); BAR; SCHED;
                STAGE(SB(1, 1), b3 + HTB);
                WAIT_V(6); BAR; MMA(1, 1, At, B1); BAR;
            }
            {
                asm volatile("s_nop 15\n\ts_nop 15" ::: "memory");
                bad = (unsigned)__builtin_amdgcn_readfirstlane(__hip_atomic_load(tmo, __ATOMIC_RELAXED, __HIP_MEMORY_SCOPE_AGENT)) == call;
                const float qn = __builtin_nanf("");
                const float scl = bad ? qn : sc;
                const int row0 = ti * 256 + wr * 64 + fr, col0 = tj * 256 + wc * 64 + 4 * fq;
#pragma unroll
                for (int ai = 0; ai < 2; ++ai)
#pragma unroll
                    for (int m = 0; m < 4; ++m) { float* rowp = p.out + (size_t)(row0 + ai * 128 + m * 16) * N + col0;
#pragma unroll
                        for (int r = 0; r < 4; ++r) { f32x4 v = (f32x4){acc[ai][0][m][0][r], acc[ai][0][m][1][r], acc[ai][1][m][0][r], acc[ai][1][m][1][r]} * scl;
                            if (!bad) { v[0] = fmaxf(v[0], 0.f); v[1] = fmaxf(v[1], 0.f); v[2] = fmaxf(v[2], 0.f); v[3] = fmaxf(v[3], 0.f); }
                            *(f32x4*)(rowp + r * 16) = v; } }
            }
            if (!has_next) break;
            ZERO_ACC();
            u = un; ti = nti; pair = npair; tj = ntj; cA = nA; cB = nB; first = false;
        }
        WAIT_V(0);
        if (wr == 0) BAR;
        BAR;
        }
    }
    __syncthreads();
    if (tid == 0) {
        const unsigned t = __hip_atomic_fetch_add((unsigned*)(myws + WS_DONE), 1u, __ATOMIC_RELAXED, __HIP_MEMORY_SCOPE_AGENT);
        if (t == call * GRID - 1u) __hip_atomic_store((unsigned*)(myws + WS_CALL), call, __ATOMIC_RELAXED, __HIP_MEMORY_SCOPE_AGENT);
    }
}
}

extern "C" void kernel_launch(int rank, int nranks, ncclComm_t comm, hipStream_t stream,
                              void* const* d_in, const int* in_sizes, int n_in,
                              void* d_out, int out_size,
                              const int* in_dtypes, int out_dtype,
                              void* const* d_out_all,
                              void* const* d_ws_all, size_t ws_bytes) {
    if (nranks != NR || ws_bytes < WS_NEED || n_in < 4) { fprintf(stderr, "unexpected configuration\n"); return; }
    HIP_CHECK(hipFuncSetAttribute((const void*)fused_k, hipFuncAttributeMaxDynamicSharedMemorySize, LDS_BYTES));
    Params p;
    p.x = (const u8*)d_in[0]; p.w = (const u8*)d_in[1]; p.sx = (const float*)d_in[2]; p.sw = (const float*)d_in[3];
    p.out = (float*)d_out; p.rank = rank;
    for (int i = 0; i < NR; ++i) p.ws[i] = (u8*)d_ws_all[i];
    hipLaunchKernelGGL(fused_k, dim3(GRID), dim3(512), LDS_BYTES, stream, p);
    HIP_CHECK(hipGetLastError());
}
```

```cpp
#include <hip/hip_runtime.h>
#include <rccl/rccl.h>
#include <cstdio>

#define HIP_CHECK(expr) do { hipError_t _e = (expr); if (_e != hipSuccess) { \
    fprintf(stderr, "HIP error %s at %s:%d\n", hipGetErrorString(_e), __FILE__, __LINE__); \
    return; } } while (0)

namespace {
#define LAS __attribute__((address_space(3)))
typedef int i32x4 __attribute__((ext_vector_type(4)));
typedef int i32x8 __attribute__((ext_vector_type(8)));
typedef float f32x4 __attribute__((ext_vector_type(4)));
typedef unsigned u32x4 __attribute__((ext_vector_type(4)));
typedef unsigned char u8;

constexpr int M = 4096, N = 16384, K = 4096, NR = 8, KS = K / NR;
constexpr int HTB = 16384, BLK = 32768, STAGE_BYTES = 8 * HTB, LDS_BYTES = 163840, LKNOWN_OFF = LDS_BYTES - 64;
constexpr int NT = 32, NI = 16, NJ = 64;
#define CSTEP 16
#define RAUX 17
#define POLL_NUM 64
#define SWIN 2
constexpr int GRID = 256, C_WG = 32, G_WG = GRID - C_WG;
constexpr size_t WS_CALL = 0, WS_DONE = 128, WS_TMO = 256, WS_LOCAL = 384, WS_CLAIM = 512;
constexpr size_t WS_FLAGS = 65536;
constexpr size_t WS_DATA = 1 << 20;
constexpr size_t A_BYTES = (size_t)M * K, B_BYTES = (size_t)K * N, PAR_BYTES = A_BYTES + B_BYTES;
constexpr size_t WS_NEED = WS_DATA + 2 * PAR_BYTES;
constexpr unsigned long long DEADLINE = 50000000ull;
constexpr unsigned PROG_MUL = 4096;

struct Params { const u8* x; const u8* w; const float* sx; const float* sw; float* out; u8* ws[NR]; int rank; };

__device__ __forceinline__ int lds_byte(int r, int c) { const int st = (r >> 4) * 2 + (c >> 6), ob = (r & 15) * 64 + (c & 63); return st * 1024 + (ob ^ (((ob >> 9) & 1) << 5)); }
__host__ __device__ __forceinline__ bool is_comm(int bx) { return bx < C_WG; }

#define STAMP(idx) do {} while (0)
#define WAIT_V(n) asm volatile("s_waitcnt vmcnt(" #n ")" ::: "memory")
#define WAIT_L(n) asm volatile("s_waitcnt lgkmcnt(" #n ")" ::: "memory")
#define BAR __builtin_amdgcn_s_barrier()
#define SCHED __builtin_amdgcn_sched_barrier(0)

constexpr int HCOLS = 14, HSTEP = 128 + HCOLS * 8, U_HEAD = 4 * HSTEP, RSTEP = HCOLS * 8, U_LEFT = U_HEAD + 3 * 4 * RSTEP, LCOLS = NJ - 4 * HCOLS, U_TOTAL = U_LEFT + 4 * LCOLS * 8;
static_assert(U_TOTAL == 2560 && LCOLS == 8 && G_WG == 16 * HCOLS, "schedule");
struct UnitDesc { int isx, kk, a, b; };
__device__ __forceinline__ UnitDesc unit_desc(int u) {
    UnitDesc d;
    if (u < U_HEAD) { const int kk = u / HSTEP, r = u - kk * HSTEP; d.kk = kk;
        if (r < 128) { d.isx = 1; d.a = r >> 3; d.b = r & 7; }
        else { const int q = r - 128; d.isx = 0; d.a = q >> 3; d.b = q & 7; }
    } else if (u < U_LEFT) { const int v = u - U_HEAD, g = v / RSTEP, r = v - g * RSTEP; d.isx = 0; d.kk = g & 3; d.a = HCOLS * (1 + (g >> 2)) + (r >> 3); d.b = r & 7; }
    else { const int v = u - U_LEFT; d.isx = 0; d.kk = v >> 6; d.a = 4 * HCOLS + ((v >> 3) & 7); d.b = v & 7; }
    return d;
}
__device__ __forceinline__ int uneed(int j, int kk) {
    if (j < HCOLS) return kk * HSTEP + 128 + (j + 1) * 8;
    if (j < 4 * HCOLS) { const int r = j / HCOLS; return U_HEAD + ((r - 1) * 4 + kk) * RSTEP + (j - r * HCOLS + 1) * 8; }
    return U_LEFT + kk * (LCOLS * 8) + (j - 4 * HCOLS + 1) * 8;
}
__device__ __forceinline__ void unit_load_lds(const Params& p, int u, LAS u8* dst, int lane) {
    const UnitDesc d = unit_desc(u);
    const int s = lane >> 2, cqp = lane & 3;
    if (d.isx) {
        const int cq = cqp ^ ((s >> 3) << 1);
        const u8* src = p.x + (size_t)(d.a * 256 + (d.b >> 2) * 128 + (d.b & 3) * 32 + s) * KS + d.kk * 128 + cq * 16;
#pragma unroll
        for (int i = 0; i < 4; ++i) __builtin_amdgcn_global_load_lds((const unsigned*)(src + (size_t)((i >> 1) * 16) * KS + (i & 1) * 64), (LAS unsigned*)(dst + i * 1024), 16, 0, 0);
    } else {
        const int wc = d.b >> 1, kh = d.b & 1;
        const u8* src = p.w + (size_t)(d.kk * 128 + kh * 64 + s) * N + d.a * 256 + wc * 64 + cqp * 16;
#pragma unroll
        for (int i = 0; i < 4; ++i) __builtin_amdgcn_global_load_lds((const unsigned*)(src + (size_t)(i * 16) * N), (LAS unsigned*)(dst + i * 1024), 16, 0, 0);
    }
}
__device__ __forceinline__ void unit_transform(bool isx, const LAS u8* tl, int lane, u32x4 (&cur)[4]) {
    if (isx) {
#pragma unroll
        for (int i = 0; i < 4; ++i) cur[i] = *(const LAS u32x4*)(tl + i * 1024 + lane * 16);
    } else {
        const int s = lane >> 2, cqp = lane & 3;
        const int cq = cqp ^ ((s >> 3) << 1);
        const int rd_off = (cq * 16) * 64 + s * 4;
        unsigned ld[16];
#pragma unroll
        for (int i = 0; i < 16; ++i) ld[i] = *(const LAS unsigned*)(tl + rd_off + i * 64);
#pragma unroll
        for (int t = 0; t < 4; ++t) {
            const unsigned d0 = ld[4 * t], d1 = ld[4 * t + 1], d2 = ld[4 * t + 2], d3 = ld[4 * t + 3];
            const unsigned a = __builtin_amdgcn_perm(d1, d0, 0x05010400u), b = __builtin_amdgcn_perm(d1, d0, 0x07030602u);
            const unsigned c = __builtin_amdgcn_perm(d3, d2, 0x05010400u), e = __builtin_amdgcn_perm(d3, d2, 0x07030602u);
            cur[0][t] = __builtin_amdgcn_perm(c, a, 0x05040100u); cur[1][t] = __builtin_amdgcn_perm(c, a, 0x07060302u);
            cur[2][t] = __builtin_amdgcn_perm(e, b, 0x05040100u); cur[3][t] = __builtin_amdgcn_perm(e, b, 0x07060302u);
        }
    }
}
__device__ __forceinline__ unsigned unit_dst(const UnitDesc& d, int rank, int i, int lane) {
    if (d.isx) return (unsigned)(((d.kk * 8 + rank) * NI + d.a) * BLK + d.b * 4096 + i * 1024 + lane * 16);
    const int wc = d.b >> 1, kh = d.b & 1;
    return (unsigned)(A_BYTES + (size_t)((d.kk * 8 + rank) * NJ + d.a) * BLK + (i >> 1) * HTB + ((wc * 2 + (i & 1)) * 2 + kh) * 1024 + lane * 16);
}

#define RING 2
#define PRE 2
#define CLD 2
constexpr int CL_RING = 0, CL_RAW = RING * 4096, CL_CTL = CL_RAW + (PRE + 1) * 4096;
static_assert(CL_CTL + 256 <= LKNOWN_OFF, "comm LDS");
__device__ __forceinline__ void comm_role(const Params& p, unsigned call, int cwg, int wid, LAS u8* lds) {
    const int lane = threadIdx.x & 63;
    const int rank = p.rank;
    volatile LAS int* ctl = (volatile LAS int*)(lds + CL_CTL);
    if (threadIdx.x < 9) ctl[threadIdx.x] = threadIdx.x == 8 ? 0x7fffffff : 0;
    __syncthreads();
    if (wid == 0) {
        __builtin_amdgcn_s_setprio(3);
        unsigned* claim = (unsigned*)(p.ws[rank] + WS_CLAIM);
        const unsigned cbase = (call - 1u) * (unsigned)(U_TOTAL + C_WG * CLD);
        unsigned long long sexec;
        asm volatile("v_accvgpr_write_b32 a3, 1\n\ts_nop 1" ::: "a3");
#define CLAIM(reg) asm volatile("s_mov_b64 %0, exec\n\ts_mov_b64 exec, 1\n\tglobal_atomic_add a" #reg ", %1, a3, off sc0\n\ts_mov_b64 exec, %0" : "=&s"(sexec) : "v"(claim) : "memory", "a0", "a1", "a2", "a3")
        CLAIM(0); if (CLD > 1) CLAIM(1); if (CLD > 2) CLAIM(2);
        int j = 0, nl = 0x7fffffff;
        int lslot = 0, pslot = 0;
        bool fin = false;
#define STEP(reg) do { \
            if (nl == 0x7fffffff) { \
                if (j < CLD) WAIT_V(0); else asm volatile("s_waitcnt vmcnt(%0)" :: "n"(5 * (CLD - 1)) : "memory"); \
                unsigned cv; asm volatile("v_accvgpr_read_b32 %0, a" #reg : "=v"(cv)); \
                const int id = (int)((unsigned)__builtin_amdgcn_readfirstlane(cv) - cbase); \
                if (id < U_TOTAL) { unit_load_lds(p, id, lds + CL_RAW + lslot * 4096, lane); if (lane == 0) ctl[16 + lslot] = id; CLAIM(reg); if (++lslot > PRE) lslot = 0; } \
                else { nl = j; if (lane == 0) ctl[8] = nl; } \
            } \
            const int k = j - PRE; \
            if (k >= 0) { \
                if (k >= nl) { fin = true; } else { \
                    if (nl == 0x7fffffff) asm volatile("s_waitcnt vmcnt(%0)" :: "n"(5 * PRE + 1) : "memory"); else WAIT_V(0); \
                    const int id = __builtin_amdgcn_readfirstlane(ctl[16 + pslot]); \
                    u32x4 cur[4]; \
                    unit_transform(unit_desc(id).isx, lds + CL_RAW + pslot * 4096, lane, cur); \
                    if (k >= RING) { for (;;) { int mn = ctl[1 + (lane % 7)]; \
                            _Pragma("unroll") for (int o = 4; o; o >>= 1) { const int x = __shfl_xor(mn, o); mn = x < mn ? x : mn; } \
                            if (__builtin_amdgcn_readfirstlane(mn) > k - RING) break; \
                            __builtin_amdgcn_s_sleep(1); } } \
                    LAS u8* rg = lds + CL_RING + (k % RING) * 4096 + lane * 16; \
                    _Pragma("unroll") for (int i = 0; i < 4; ++i) *(LAS u32x4*)(rg + i * 1024) = cur[i]; \
                    if (lane == 0) ctl[32 + (k % RING)] = id; \
                    WAIT_L(0); \
                    if (lane == 0) ctl[0] = k + 1; \
                    if (++pslot > PRE) pslot = 0; \
                } } \
            ++j; } while (0)
        while (!fin) { STEP(0); if (fin) break; if (CLD > 1) { STEP(1); if (fin) break; } if (CLD > 2) STEP(2); }
        WAIT_V(0);
        __builtin_amdgcn_s_setprio(0);
    } else {
        const int dest = (rank + wid) & 7;
        u8* dws = p.ws[dest];
        const __amdgpu_buffer_rsrc_t rs = __builtin_amdgcn_make_buffer_rsrc(dws + WS_DATA + (size_t)(call & 1) * PAR_BYTES, 0, (int)PAR_BYTES, 0x00020000);
        unsigned* prog = (unsigned*)(dws + WS_FLAGS) + rank * 32 + cwg;
        const unsigned pbase = call * PROG_MUL;
        int hist[SWIN];
#pragma unroll
        for (int i = 0; i < SWIN; ++i) hist[i] = -1;
        for (int k = 0;; ++k) {
            bool end = false;
            while (__builtin_amdgcn_readfirstlane(ctl[0]) <= k) { if (__builtin_amdgcn_readfirstlane(ctl[8]) <= k) { end = true; break; } __builtin_amdgcn_s_sleep(1); }
            if (end) break;
            if (__builtin_amdgcn_readfirstlane(ctl[0]) - k >= 3) __builtin_amdgcn_s_setprio(2); else __builtin_amdgcn_s_setprio(0);
            const LAS u8* rg = lds + CL_RING + (k % RING) * 4096 + lane * 16;
            u32x4 cur[4];
#pragma unroll
            for (int i = 0; i < 4; ++i) cur[i] = *(const LAS u32x4*)(rg + i * 1024);
            const int id = __builtin_amdgcn_readfirstlane(ctl[32 + (k % RING)]);
            WAIT_L(0);
            if (lane == 0) ctl[wid] = k + 1;
            const UnitDesc d = unit_desc(id);
#pragma unroll
            for (int i = 0; i < 4; ++i) __builtin_amdgcn_raw_buffer_store_b128(cur[i], rs, unit_dst(d, rank, i, lane), 0, RAUX);
            asm volatile("s_waitcnt vmcnt(%0)" :: "n"(4 * SWIN) : "memory");
            const int oldest = k >= SWIN - 1 ? hist[SWIN - 2 >= 0 ? SWIN - 2 : 0] : (k == 0 ? id : -1);
#pragma unroll
            for (int i = SWIN - 1; i > 0; --i) hist[i] = hist[i - 1];
            hist[0] = id;
            const int X = SWIN == 1 ? id : oldest;
            if (X >= 0 && lane == 0) __hip_atomic_store(prog, pbase + (unsigned)X, __ATOMIC_RELAXED, __HIP_MEMORY_SCOPE_SYSTEM);
        }
        __builtin_amdgcn_s_setprio(0);
        WAIT_V(0);
        if (lane == 0) __hip_atomic_store(prog, pbase + (unsigned)(PROG_MUL - 1), __ATOMIC_RELAXED, __HIP_MEMORY_SCOPE_SYSTEM);
    }
}

#define SA(b, h) (((b) * 2 + (h)) * HTB)
#define SB(b, h) ((4 + (b) * 2 + (h)) * HTB)
#define GLDS(gptr, ldsoff) do { const char* _g = (const char*)(gptr); asm volatile("" : "+s"(_g)); \
        __builtin_amdgcn_global_load_lds((const unsigned*)(_g + voff), (LAS unsigned*)(lds + (ldsoff) + ldsw), 16, 0, 0); } while (0)
#define STAGE(bufoff, gbase) do { GLDS((const char*)(gbase), (bufoff)); GLDS((const char*)(gbase) + 8192, (bufoff) + 8192); } while (0)
#define LDF(off) __builtin_shufflevector(*(const LAS i32x4*)(lds + (off)), *(const LAS i32x4*)(lds + (off) + 1024), 0, 1, 2, 3, 4, 5, 6, 7)
#define LDA(dst, b, h) do { _Pragma("unroll") for (int m = 0; m < 4; ++m) dst[m] = LDF(SA(b, h) + aoff + m * 2048); } while (0)
#define LDB(dst, b, h) do { _Pragma("unroll") for (int n = 0; n < 2; ++n) dst[n] = LDF(SB(b, h) + boff + n * 2048); } while (0)
#define C10(p) "a" #p "0","a" #p "1","a" #p "2","a" #p "3","a" #p "4","a" #p "5","a" #p "6","a" #p "7","a" #p "8","a" #p "9"
#define ACC_CLOB "a0","a1","a2","a3","a4","a5","a6","a7","a8","a9", C10(1),C10(2),C10(3),C10(4),C10(5),C10(6),C10(7),C10(8),C10(9),C10(10),C10(11),"a120","a121","a122","a123","a124","a125","a126","a127"
#define MF(o, A, B) "v_mfma_f32_16x16x128_f8f6f4 a[%c6+" #o ":%c6+" #o "+3], " A ", " B ", a[%c6+" #o ":%c6+" #o "+3]\n\t"
#define MFZ(o, A, B) "v_mfma_f32_16x16x128_f8f6f4 a[%c6+" #o ":%c6+" #o "+3], " A ", " B ", 0\n\t"
#define MMA(ai, bj, At, Bt) asm volatile("s_setprio 1\n\t" MF(0, "%0", "%4") MF(4, "%0", "%5") MF(8, "%1", "%4") MF(12, "%1", "%5") MF(16, "%2", "%4") MF(20, "%2", "%5") MF(24, "%3", "%4") MF(28, "%3", "%5") "s_setprio 0" \
        :: "v"(At[0]), "v"(At[1]), "v"(At[2]), "v"(At[3]), "v"(Bt[0]), "v"(Bt[1]), "i"(((ai) * 2 + (bj)) * 32) : ACC_CLOB)
#define MMAZ(ai, bj, At, Bt) asm volatile("s_setprio 1\n\t" MFZ(0, "%0", "%4") MFZ(4, "%0", "%5") MFZ(8, "%1", "%4") MFZ(12, "%1", "%5") MFZ(16, "%2", "%4") MFZ(20, "%2", "%5") MFZ(24, "%3", "%4") MFZ(28, "%3", "%5") "s_setprio 0" \
        :: "v"(At[0]), "v"(At[1]), "v"(At[2]), "v"(At[3]), "v"(Bt[0]), "v"(Bt[1]), "i"(((ai) * 2 + (bj)) * 32) : ACC_CLOB)
#define MM(Z, ai, bj, At, Bt) do { if (Z) MMAZ(ai, bj, At, Bt); else MMA(ai, bj, At, Bt); } while (0)
#define ACC_RD4(v, i0, i1, i2, i3) asm volatile("v_accvgpr_read_b32 %0, a[%c4]\n\tv_accvgpr_read_b32 %1, a[%c5]\n\tv_accvgpr_read_b32 %2, a[%c6]\n\tv_accvgpr_read_b32 %3, a[%c7]" \
        : "=v"((v)[0]), "=v"((v)[1]), "=v"((v)[2]), "=v"((v)[3]) : "i"(i0), "i"(i1), "i"(i2), "i"(i3))
#define EPI1(ai, m, r) do { float x_[4]; ACC_RD4(x_, (ai) * 64 + (m) * 8 + (r), (ai) * 64 + (m) * 8 + 4 + (r), (ai) * 64 + 32 + (m) * 8 + (r), (ai) * 64 + 32 + (m) * 8 + 4 + (r)); \
        f32x4 v = (f32x4){x_[0], x_[1], x_[2], x_[3]} * scl; \
        v[0] = v[0] < 0.f ? 0.f : v[0]; v[1] = v[1] < 0.f ? 0.f : v[1]; v[2] = v[2] < 0.f ? 0.f : v[2]; v[3] = v[3] < 0.f ? 0.f : v[3]; \
        __builtin_amdgcn_raw_buffer_store_b128(__builtin_bit_cast(u32x4, v), rso, eoff + (unsigned)(((ai) * 128 + (m) * 16 + (r)) * N * 4), 0, 0); } while (0)
#define EPIM(ai, m) do { EPI1(ai, m, 0); EPI1(ai, m, 1); EPI1(ai, m, 2); EPI1(ai, m, 3); } while (0)

__global__ __launch_bounds__(512, 2) void fused_k(Params p) {
    extern __shared__ __attribute__((aligned(16))) u8 lds_raw[];
    LAS u8* lds = (LAS u8*)lds_raw;
    const int tid = threadIdx.x, wid = __builtin_amdgcn_readfirstlane(tid >> 6);
    const int bx = blockIdx.x, rank = p.rank;
    u8* myws = p.ws[rank];
    const unsigned call = (unsigned)__builtin_amdgcn_readfirstlane(__hip_atomic_load((unsigned*)(myws + WS_CALL), __ATOMIC_RELAXED, __HIP_MEMORY_SCOPE_AGENT)) + 1u;
    unsigned* tmo = (unsigned*)(myws + WS_TMO);
    const int wr = wid >> 2, wc = wid & 3;
    const unsigned ldsw = (unsigned)wid * 1024u;
    volatile LAS int* lknown = (volatile LAS int*)(lds + LKNOWN_OFF);
    const size_t par = WS_DATA + (size_t)(call & 1) * PAR_BYTES;
    const char* Abase = (const char*)myws + par;
    const char* Bbase = Abase + A_BYTES;
    const unsigned* flags = (const unsigned*)(myws + WS_FLAGS);
    const float sc = p.sx[0] * p.sw[0];
    int known = 0;
    bool bad = false;
    auto ready = [&](int need) {
        if (known >= need) return;
        if (wid == 0) {
            const int lane = threadIdx.x & 63, c = lane & 31;
            const unsigned pbase = call * PROG_MUL;
            const unsigned long long t0 = __builtin_amdgcn_s_memrealtime();
            int kn;
            for (;;) {
                unsigned mn = 0x7fffffffu;
                unsigned v[4];
#pragma unroll
                for (int i = 0; i < 4; ++i) { const int src = (lane >> 5) + 2 * i;
                    v[i] = src == rank ? __hip_atomic_load((unsigned*)(myws + WS_LOCAL), __ATOMIC_RELAXED, __HIP_MEMORY_SCOPE_AGENT)
                                       : __hip_atomic_load(flags + src * 32 + c, __ATOMIC_RELAXED, __HIP_MEMORY_SCOPE_SYSTEM); }
#pragma unroll
                for (int i = 0; i < 4; ++i) { const int src = (lane >> 5) + 2 * i; unsigned pre;
                    if (src == rank) pre = v[i] >= call * (unsigned)G_WG ? 0x7fffffffu : 0u;
                    else pre = v[i] >= pbase ? v[i] - pbase : 0u;
                    mn = pre < mn ? pre : mn; }
#pragma unroll
                for (int o = 32; o; o >>= 1) { const unsigned x = (unsigned)__shfl_xor((int)mn, o); mn = x < mn ? x : mn; }
                kn = __builtin_amdgcn_readfirstlane((int)mn);
                if (kn >= need) break;
                if ((unsigned)__builtin_amdgcn_readfirstlane(__hip_atomic_load(tmo, __ATOMIC_RELAXED, __HIP_MEMORY_SCOPE_AGENT)) == call || __builtin_amdgcn_s_memrealtime() - t0 > DEADLINE) {
                    if (lane == 0) __hip_atomic_store(tmo, call, __ATOMIC_RELAXED, __HIP_MEMORY_SCOPE_AGENT);
                    kn = 0x7fffffff; break; }
                { const unsigned long long w0 = __builtin_amdgcn_s_memrealtime(); const unsigned long long dt = (unsigned long long)((need - kn) * POLL_NUM / 16 + 30);
                  while (__builtin_amdgcn_s_memrealtime() - w0 < dt) __builtin_amdgcn_s_sleep(8); }
            }
            __builtin_amdgcn_fence(__ATOMIC_ACQUIRE, "");
            if (lane == 0) *lknown = kn;
            asm volatile("s_waitcnt vmcnt(0) lgkmcnt(0)" ::: "memory");
        }
        asm volatile("" ::: "memory"); BAR; asm volatile("" ::: "memory");
        known = __builtin_amdgcn_readfirstlane(*lknown);
        asm volatile("s_waitcnt lgkmcnt(0)" ::: "memory");
        if (known == 0x7fffffff) bad = true;
    };
    auto kt = [&](int n) { return (n & ~7) | ((rank + n) & 7); };
#define APTR(base, n) ((base) + (size_t)kt(n) * (NI * BLK))
#define BPTR(base, n) ((base) + (size_t)kt(n) * (NJ * BLK))
    i32x8 At[4], B0[2], B1[2];
    int sidx = 1; (void)sidx;

    STAMP(0);
    if (is_comm(bx)) {
        comm_role(p, call, bx, wid, lds);
        STAMP(1);
    } else {
        const int lane = tid & 63, fr = lane & 15, fq = lane >> 4;
        const unsigned voff = (unsigned)tid * 16u;
        const int aoff = lds_byte(wr * 64 + fr, fq * 16), boff = lds_byte(wc * 32 + fr, fq * 16);
        const int xcd = bx & 7, loc = (bx >> 3) - C_WG / 8;
        {
            const __amdgpu_buffer_rsrc_t rsl = __builtin_amdgcn_make_buffer_rsrc(myws + par, 0, (int)PAR_BYTES, 0x00020000);
            constexpr int NGW = G_WG * 8;
            static_assert(2 * NGW >= U_TOTAL, "prologue");
            const int gw = (bx - C_WG) * 8 + wid;
            const bool l1 = gw + NGW < U_TOTAL;
            LAS u8* tl = lds + wid * 8192;
            unit_load_lds(p, gw, tl, lane);
            if (l1) unit_load_lds(p, gw + NGW, tl + 4096, lane);
            WAIT_V(0);
            { const UnitDesc d0 = unit_desc(gw); u32x4 c0[4]; unit_transform(d0.isx, tl, lane, c0);
#pragma unroll
              for (int i = 0; i < 4; ++i) __builtin_amdgcn_raw_buffer_store_b128(c0[i], rsl, unit_dst(d0, rank, i, lane), 0, 16); }
            if (l1) { const UnitDesc d1 = unit_desc(gw + NGW); u32x4 c1[4]; unit_transform(d1.isx, tl + 4096, lane, c1);
#pragma unroll
                for (int i = 0; i < 4; ++i) __builtin_amdgcn_raw_buffer_store_b128(c1[i], rsl, unit_dst(d1, rank, i, lane), 0, 16); }
            asm volatile("s_waitcnt vmcnt(0) lgkmcnt(0)" ::: "memory");
            __syncthreads();
            if (tid == 0) __hip_atomic_fetch_add((unsigned*)(myws + WS_LOCAL), 1u, __ATOMIC_RELAXED, __HIP_MEMORY_SCOPE_AGENT);
            STAMP(20);
        }
        const int ti = 2 * xcd + (loc & 1);
        int tj = loc >> 1, gate = (loc >> 1) < 7 ? 6 : 13;
        const char* cA = Abase + (size_t)ti * BLK;
        const char* cB = Bbase + (size_t)tj * BLK;
        ready(uneed(gate, 0));
        STAMP(sidx); ++sidx;
        {
            const char* a0 = APTR(cA, 0); const char* b0 = BPTR(cB, 0); const char* a1 = APTR(cA, 1); const char* b1 = BPTR(cB, 1);
            STAGE(SB(0, 0), b0); STAGE(SA(0, 0), a0); STAGE(SB(0, 1), b0 + HTB); STAGE(SA(0, 1), a0 + HTB);
            if (wr == 1) BAR;
            WAIT_V(4); BAR;
            STAGE(SB(1, 0), b1); STAGE(SA(1, 0), a1); STAGE(SB(1, 1), b1 + HTB);
            WAIT_V(6); BAR;
        }
        bool first = true; (void)first;
#define ITER(MODE) do { \
                LDB(B0, 0, 0); SCHED; LDA(At, 0, 0); STAGE(SA(1, 1), a1 + HTB); \
                if (MODE == 1) { SCHED; EPIM(1, 0); EPIM(1, 1); SCHED; } \
                WAIT_L(8); BAR; WAIT_L(0); MM(MODE == 1, 0, 0, At, B0); BAR; SCHED; \
                LDB(B1, 0, 1); STAGE(SB(0, 0), b2); \
                if (MODE == 1) { SCHED; EPIM(1, 2); EPIM(1, 3); SCHED; } \
                BAR; WAIT_L(0); MM(MODE == 1, 0, 1, At, B1); BAR; \
                LDA(At, 0, 1); STAGE(SA(0, 0), a2); \
                BAR; WAIT_L(0); MM(MODE == 1, 1, 0, At, B0); BAR; SCHED; \
                STAGE(SB(0, 1), b2 + HTB); \
                if (MODE == 1) WAIT_V(22); else WAIT_V(6); \
                BAR; MM(MODE == 1, 1, 1, At, B1); BAR; \
                LDB(B0, 1, 0); SCHED; LDA(At, 1, 0); STAGE(SA(0, 1), a2 + HTB); \
                WAIT_L(8); BAR; WAIT_L(0); MMA(0, 0, At, B0); BAR; SCHED; \
                LDB(B1, 1, 1); STAGE(SB(1, 0), b3); \
                BAR; WAIT_L(0); MMA(0, 1, At, B1); BAR; \
                LDA(At, 1, 1); STAGE(SA(1, 0), a3); \
                if (MODE == 2) { SCHED; asm volatile("s_nop 7"); EPIM(0, 0); EPIM(0, 1); SCHED; } \
                BAR; WAIT_L(0); MMA(1, 0, At, B0); BAR; SCHED; \
                STAGE(SB(1, 1), b3 + HTB); \
                if (MODE == 2) { SCHED; EPIM(0, 2); EPIM(0, 3); SCHED; WAIT_V(22); } else WAIT_V(6); \
                BAR; MMA(1, 1, At, B1); BAR; } while (0)
        const unsigned eoff0 = (unsigned)(((ti * 256 + wr * 64 + 4 * fq) * N + wc * 64 + 4 * fr) * 4);
        __amdgpu_buffer_rsrc_t rso = __builtin_amdgcn_make_buffer_rsrc(p.out, 0, 0, 0x00020000);
        unsigned eoff = eoff0;
        float scl = sc;
        for (;;) {
            const bool has_next = tj + HCOLS < 4 * HCOLS;
            const int ntj = tj + HCOLS, ngate = gate + HCOLS;
            const char* nB = has_next ? Bbase + (size_t)ntj * BLK : cB;
            {
                const char* a1 = APTR(cA, 1); const char* a2 = APTR(cA, 2); const char* b2 = BPTR(cB, 2); const char* a3 = APTR(cA, 3); const char* b3 = BPTR(cB, 3);
                asm volatile("s_nop 7");
                ITER(1);
            }
            for (int t = 2; t < NT - 2; t += 2) {
                if (((t + 2) & 7) == 0) { if (first) { STAMP(sidx); ++sidx; } ready(uneed(gate, (t + 2) >> 3)); if (first) { STAMP(sidx); ++sidx; } }
                const char* a1 = APTR(cA, t + 1); const char* a2 = APTR(cA, t + 2); const char* b2 = BPTR(cB, t + 2); const char* a3 = APTR(cA, t + 3); const char* b3 = BPTR(cB, t + 3);
                ITER(0);
            }
            {
                if (has_next) ready(uneed(ngate, 0));
                rso = __builtin_amdgcn_make_buffer_rsrc(p.out, 0, M * N * 4, 0x00020000);
                eoff = eoff0 + (unsigned)tj * 1024u;
                scl = bad ? __builtin_nanf("") : sc;
                const char* a1 = APTR(cA, NT - 1); const char* a2 = APTR(cA, 0); const char* b2 = BPTR(nB, 0); const char* a3 = APTR(cA, 1); const char* b3 = BPTR(nB, 1);
                ITER(2);
            }
            STAMP(sidx); ++sidx;
            if (!has_next) break;
            gate = ngate; tj = ntj; cB = nB; first = false;
        }
        asm volatile("s_nop 15\n\ts_nop 15" ::: "memory");
        EPIM(1, 0); EPIM(1, 1); EPIM(1, 2); EPIM(1, 3);
        WAIT_V(0);
        if (wr == 0) BAR;
    }
    {
        asm volatile("s_waitcnt vmcnt(0) lgkmcnt(0)" ::: "memory");
        BAR;
        int tz = threadIdx.x; asm volatile("" : "+v"(tz));
        const int lane = tz & 63, fr = lane & 15, fq = lane >> 4;
        const unsigned voff = (unsigned)tz * 16u;
        const int aoff = lds_byte(wr * 64 + fr, fq * 16), boff = lds_byte(wc * 32 + fr, fq * 16);
        const int tj = 4 * HCOLS + ((bx >> 3) & 7), ti = 2 * (bx & 7) + ((bx >> 6) & 1), ah = bx >> 7;
        const char* cA = Abase + (size_t)ti * BLK + ah * HTB;
        const char* cB = Bbase + (size_t)tj * BLK;
        ready(uneed(tj, 3));
        constexpr int HB = 49152;
        { const char* a = APTR(cA, 0); const char* b = BPTR(cB, 0); STAGE(0, a); STAGE(HTB, b); STAGE(2 * HTB, b + HTB);
          a = APTR(cA, 1); b = BPTR(cB, 1); STAGE(HB, a); STAGE(HB + HTB, b); STAGE(HB + 2 * HTB, b + HTB); }
        if (wr == 1) BAR;
        WAIT_V(6); BAR; BAR;
        int bo = 0;
        for (int t = 0; t < NT; ++t) {
            int nbo = bo + 2 * HB; if (nbo >= 3 * HB) nbo -= 3 * HB;
            const bool more = t + 2 < NT;
            const char* a = APTR(cA, (more ? t + 2 : t)); const char* b = BPTR(cB, (more ? t + 2 : t));
#pragma unroll
            for (int n = 0; n < 2; ++n) B0[n] = LDF(bo + HTB + boff + n * 2048);
            SCHED;
#pragma unroll
            for (int m = 0; m < 4; ++m) At[m] = LDF(bo + aoff + m * 2048);
            if (more) { STAGE(nbo, a); GLDS(b, nbo + HTB); }
            BAR; WAIT_L(0); MM(t == 0, 0, 0, At, B0); BAR; SCHED;
#pragma unroll
            for (int n = 0; n < 2; ++n) B1[n] = LDF(bo + 2 * HTB + boff + n * 2048);
            if (more) { GLDS(b + 8192, nbo + HTB + 8192); STAGE(nbo + 2 * HTB, b + HTB); WAIT_V(6); } else WAIT_V(0);
            BAR; WAIT_L(0); MM(t == 0, 0, 1, At, B1); BAR; SCHED;
            bo += HB; if (bo >= 3 * HB) bo = 0;
        }
        if (wr == 0) BAR;
        asm volatile("s_nop 15\n\ts_nop 15" ::: "memory");
        const float scl = bad ? __builtin_nanf("") : sc;
        const __amdgpu_buffer_rsrc_t rso = __builtin_amdgcn_make_buffer_rsrc(p.out, 0, M * N * 4, 0x00020000);
        const unsigned eoff = (unsigned)(((ti * 256 + ah * 128 + wr * 64 + 4 * fq) * N + tj * 256 + wc * 64 + 4 * fr) * 4);
        EPIM(0, 0); EPIM(0, 1); EPIM(0, 2); EPIM(0, 3);
        STAMP(sidx); ++sidx;
    }
    __syncthreads();
    if (tid == 0) {
        const unsigned t = __hip_atomic_fetch_add((unsigned*)(myws + WS_DONE), 1u, __ATOMIC_RELAXED, __HIP_MEMORY_SCOPE_AGENT);
        if (t == call * GRID - 1u) __hip_atomic_store((unsigned*)(myws + WS_CALL), call, __ATOMIC_RELAXED, __HIP_MEMORY_SCOPE_AGENT);
    }
}
}

extern "C" void kernel_launch(int rank, int nranks, ncclComm_t comm, hipStream_t stream,
                              void* const* d_in, const int* in_sizes, int n_in,
                              void* d_out, int out_size,
                              const int* in_dtypes, int out_dtype,
                              void* const* d_out_all,
                              void* const* d_ws_all, size_t ws_bytes) {
    if (nranks != NR || rank < 0 || rank >= NR || ws_bytes < WS_NEED || n_in < 4 || out_size != M * N) { fprintf(stderr, "unexpected configuration\n"); return; }
    HIP_CHECK(hipFuncSetAttribute((const void*)fused_k, hipFuncAttributeMaxDynamicSharedMemorySize, LDS_BYTES));
    Params p;
    p.x = (const u8*)d_in[0]; p.w = (const u8*)d_in[1]; p.sx = (const float*)d_in[2]; p.sw = (const float*)d_in[3];
    p.out = (float*)d_out; p.rank = rank;
    for (int i = 0; i < NR; ++i) p.ws[i] = (u8*)d_ws_all[i];
    hipLaunchKernelGGL(fused_k, dim3(GRID), dim3(512), LDS_BYTES, stream, p);
    HIP_CHECK(hipGetLastError());
}
```

```cpp
#include <hip/hip_runtime.h>
#include <rccl/rccl.h>
#include <cstdio>

#define HIP_CHECK(expr) do { hipError_t _e = (expr); if (_e != hipSuccess) { \
    fprintf(stderr, "HIP error %s at %s:%d\n", hipGetErrorString(_e), __FILE__, __LINE__); \
    return; } } while (0)

namespace {
#define LAS __attribute__((address_space(3)))
typedef int i32x4 __attribute__((ext_vector_type(4)));
typedef int i32x8 __attribute__((ext_vector_type(8)));
typedef float f32x4 __attribute__((ext_vector_type(4)));
typedef unsigned u32x4 __attribute__((ext_vector_type(4)));
typedef unsigned char u8;

constexpr int M = 4096, N = 16384, K = 4096, NR = 8, KS = K / NR;
constexpr int HTB = 16384, BLK = 32768, STAGE_BYTES = 8 * HTB, LDS_BYTES = 163840, LKNOWN_OFF = LDS_BYTES - 64;
constexpr int NT = 32, NI = 16, NJ = 64;
#define CSTEP 16
#define RAUX 17
#define POLL_NUM 64
#define SWIN 1
#define NCOMM 32
constexpr int GRID = 256, C_WG = NCOMM, G_WG = GRID - C_WG;
constexpr size_t WS_CALL = 0, WS_DONE = 128, WS_TMO = 256, WS_LOCAL = 384, WS_CLAIM = 512;
constexpr size_t WS_FLAGS = 65536;
constexpr size_t WS_DATA = 1 << 20;
constexpr size_t A_BYTES = (size_t)M * K, B_BYTES = (size_t)K * N, PAR_BYTES = A_BYTES + B_BYTES;
constexpr size_t WS_NEED = WS_DATA + 2 * PAR_BYTES;
constexpr unsigned long long DEADLINE = 50000000ull;
constexpr unsigned PROG_MUL = 4096;

struct Params { const u8* x; const u8* w; const float* sx; const float* sw; float* out; u8* ws[NR]; int rank; };

__device__ __forceinline__ int lds_byte(int r, int c) { const int st = (r >> 4) * 2 + (c >> 6), ob = (r & 15) * 64 + (c & 63); return st * 1024 + (ob ^ (((ob >> 9) & 1) << 5)); }
__host__ __device__ __forceinline__ bool is_comm(int bx) { return bx < C_WG; }

#define STAMP(idx) do {} while (0)
#define WAIT_V(n) asm volatile("s_waitcnt vmcnt(" #n ")" ::: "memory")
#define WAIT_L(n) asm volatile("s_waitcnt lgkmcnt(" #n ")" ::: "memory")
#define BAR __builtin_amdgcn_s_barrier()
#define SCHED __builtin_amdgcn_sched_barrier(0)

constexpr int HCOLS = G_WG / 16, HSTEP = 128 + HCOLS * 8, U_HEAD = 4 * HSTEP, RSTEP = HCOLS * 8, U_LEFT = U_HEAD + 3 * 4 * RSTEP, LCOLS = NJ - 4 * HCOLS, U_TOTAL = U_LEFT + 4 * LCOLS * 8;
static_assert(U_TOTAL == 2560 && G_WG == 16 * HCOLS && (LCOLS == 8 || LCOLS == 4), "schedule");
struct UnitDesc { int isx, kk, a, b; };
__device__ __forceinline__ UnitDesc unit_desc(int u) {
    UnitDesc d;
    if (u < U_HEAD) { const int kk = u / HSTEP, r = u - kk * HSTEP; d.kk = kk;
        if (r < 128) { d.isx = 1; d.a = r >> 3; d.b = r & 7; }
        else { const int q = r - 128; d.isx = 0; d.a = q >> 3; d.b = q & 7; }
    } else if (u < U_LEFT) { const int v = u - U_HEAD, g = v / RSTEP, r = v - g * RSTEP; d.isx = 0; d.kk = g & 3; d.a = HCOLS * (1 + (g >> 2)) + (r >> 3); d.b = r & 7; }
    else { const int v = u - U_LEFT; d.isx = 0; d.kk = v / (LCOLS * 8); d.a = 4 * HCOLS + ((v >> 3) & (LCOLS - 1)); d.b = v & 7; }
    return d;
}
__device__ __forceinline__ int uneed(int j, int kk) {
    if (j < HCOLS) return kk * HSTEP + 128 + (j + 1) * 8;
    if (j < 4 * HCOLS) { const int r = j / HCOLS; return U_HEAD + ((r - 1) * 4 + kk) * RSTEP + (j - r * HCOLS + 1) * 8; }
    return U_LEFT + kk * (LCOLS * 8) + (j - 4 * HCOLS + 1) * 8;
}
__device__ __forceinline__ void unit_load_lds(const Params& p, int u, LAS u8* dst, int lane) {
    const UnitDesc d = unit_desc(u);
    const int s = lane >> 2, cqp = lane & 3;
    if (d.isx) {
        const int cq = cqp ^ ((s >> 3) << 1);
        const u8* src = p.x + (size_t)(d.a * 256 + (d.b >> 2) * 128 + (d.b & 3) * 32 + s) * KS + d.kk * 128 + cq * 16;
#pragma unroll
        for (int i = 0; i < 4; ++i) __builtin_amdgcn_global_load_lds((const unsigned*)(src + (size_t)((i >> 1) * 16) * KS + (i & 1) * 64), (LAS unsigned*)(dst + i * 1024), 16, 0, 0);
    } else {
        const int wc = d.b >> 1, kh = d.b & 1;
        const u8* src = p.w + (size_t)(d.kk * 128 + kh * 64 + s) * N + d.a * 256 + wc * 64 + cqp * 16;
#pragma unroll
        for (int i = 0; i < 4; ++i) __builtin_amdgcn_global_load_lds((const unsigned*)(src + (size_t)(i * 16) * N), (LAS unsigned*)(dst + i * 1024), 16, 0, 0);
    }
}
__device__ __forceinline__ void unit_transform(bool isx, const LAS u8* tl, int lane, u32x4 (&cur)[4]) {
    if (isx) {
#pragma unroll
        for (int i = 0; i < 4; ++i) cur[i] = *(const LAS u32x4*)(tl + i * 1024 + lane * 16);
    } else {
        const int s = lane >> 2, cqp = lane & 3;
        const int cq = cqp ^ ((s >> 3) << 1);
        const int rd_off = (cq * 16) * 64 + s * 4;
        unsigned ld[16];
#pragma unroll
        for (int i = 0; i < 16; ++i) ld[i] = *(const LAS unsigned*)(tl + rd_off + i * 64);
#pragma unroll
        for (int t = 0; t < 4; ++t) {
            const unsigned d0 = ld[4 * t], d1 = ld[4 * t + 1], d2 = ld[4 * t + 2], d3 = ld[4 * t + 3];
            const unsigned a = __builtin_amdgcn_perm(d1, d0, 0x05010400u), b = __builtin_amdgcn_perm(d1, d0, 0x07030602u);
            const unsigned c = __builtin_amdgcn_perm(d3, d2, 0x05010400u), e = __builtin_amdgcn_perm(d3, d2, 0x07030602u);
            cur[0][t] = __builtin_amdgcn_perm(c, a, 0x05040100u); cur[1][t] = __builtin_amdgcn_perm(c, a, 0x07060302u);
            cur[2][t] = __builtin_amdgcn_perm(e, b, 0x05040100u); cur[3][t] = __builtin_amdgcn_perm(e, b, 0x07060302u);
        }
    }
}
__device__ __forceinline__ unsigned unit_dst(const UnitDesc& d, int rank, int i, int lane) {
    if (d.isx) return (unsigned)(((d.kk * 8 + rank) * NI + d.a) * BLK + d.b * 4096 + i * 1024 + lane * 16);
    const int wc = d.b >> 1, kh = d.b & 1;
    return (unsigned)(A_BYTES + (size_t)((d.kk * 8 + rank) * NJ + d.a) * BLK + (i >> 1) * HTB + ((wc * 2 + (i & 1)) * 2 + kh) * 1024 + lane * 16);
}

#define RING 2
#define PRE 2
#define CLD 1
constexpr int CL_RING = 0, CL_RAW = RING * 4096, CL_CTL = CL_RAW + (PRE + 1) * 4096;
static_assert(CL_CTL + 256 <= LKNOWN_OFF, "comm LDS");
__device__ __forceinline__ void comm_role(const Params& p, unsigned call, int cwg, int wid, LAS u8* lds) {
    const int lane = threadIdx.x & 63;
    const int rank = p.rank;
    volatile LAS int* ctl = (volatile LAS int*)(lds + CL_CTL);
    if (threadIdx.x < 9) ctl[threadIdx.x] = threadIdx.x == 8 ? 0x7fffffff : 0;
    __syncthreads();
    if (wid == 0) {
        __builtin_amdgcn_s_setprio(3);
        unsigned* claim = (unsigned*)(p.ws[rank] + WS_CLAIM);
        const unsigned cbase = (call - 1u) * (unsigned)(U_TOTAL + C_WG * CLD);
        unsigned long long sexec;
        asm volatile("v_accvgpr_write_b32 a3, 1\n\ts_nop 1" ::: "a3");
#define CLAIM(reg) asm volatile("s_mov_b64 %0, exec\n\ts_mov_b64 exec, 1\n\tglobal_atomic_add a" #reg ", %1, a3, off sc0\n\ts_mov_b64 exec, %0" : "=&s"(sexec) : "v"(claim) : "memory", "a0", "a1", "a2", "a3")
        CLAIM(0); if (CLD > 1) CLAIM(1); if (CLD > 2) CLAIM(2);
        int j = 0, nl = 0x7fffffff;
        int lslot = 0, pslot = 0;
        bool fin = false;
#define STEP(reg) do { \
            if (nl == 0x7fffffff) { \
                if (j < CLD) WAIT_V(0); else asm volatile("s_waitcnt vmcnt(%0)" :: "n"(5 * (CLD - 1)) : "memory"); \
                unsigned cv; asm volatile("v_accvgpr_read_b32 %0, a" #reg : "=v"(cv)); \
                const int id = (int)((unsigned)__builtin_amdgcn_readfirstlane(cv) - cbase); \
                if (id < U_TOTAL) { unit_load_lds(p, id, lds + CL_RAW + lslot * 4096, lane); if (lane == 0) ctl[16 + lslot] = id; CLAIM(reg); if (++lslot > PRE) lslot = 0; } \
                else { nl = j; if (lane == 0) ctl[8] = nl; } \
            } \
            const int k = j - PRE; \
            if (k >= 0) { \
                if (k >= nl) { fin = true; } else { \
                    if (nl == 0x7fffffff) asm volatile("s_waitcnt vmcnt(%0)" :: "n"(5 * PRE + 1) : "memory"); else WAIT_V(0); \
                    const int id = __builtin_amdgcn_readfirstlane(ctl[16 + pslot]); \
                    u32x4 cur[4]; \
                    unit_transform(unit_desc(id).isx, lds + CL_RAW + pslot * 4096, lane, cur); \
                    if (k >= RING) { for (;;) { int mn = ctl[1 + (lane % 7)]; \
                            _Pragma("unroll") for (int o = 4; o; o >>= 1) { const int x = __shfl_xor(mn, o); mn = x < mn ? x : mn; } \
                            if (__builtin_amdgcn_readfirstlane(mn) > k - RING) break; \
                            __builtin_amdgcn_s_sleep(1); } } \
                    LAS u8* rg = lds + CL_RING + (k % RING) * 4096 + lane * 16; \
                    _Pragma("unroll") for (int i = 0; i < 4; ++i) *(LAS u32x4*)(rg + i * 1024) = cur[i]; \
                    if (lane == 0) ctl[32 + (k % RING)] = id; \
                    WAIT_L(0); \
                    if (lane == 0) ctl[0] = k + 1; \
                    if (++pslot > PRE) pslot = 0; \
                } } \
            ++j; } while (0)
        while (!fin) { STEP(0); if (fin) break; if (CLD > 1) { STEP(1); if (fin) break; } if (CLD > 2) STEP(2); }
        WAIT_V(0);
        __builtin_amdgcn_s_setprio(0);
    } else {
        const int dest = (rank + wid) & 7;
        u8* dws = p.ws[dest];
        const __amdgpu_buffer_rsrc_t rs = __builtin_amdgcn_make_buffer_rsrc(dws + WS_DATA + (size_t)(call & 1) * PAR_BYTES, 0, (int)PAR_BYTES, 0x00020000);
        unsigned* prog = (unsigned*)(dws + WS_FLAGS) + rank * 32 + cwg;
        const unsigned pbase = call * PROG_MUL;
        int hist[SWIN];
#pragma unroll
        for (int i = 0; i < SWIN; ++i) hist[i] = -1;
        for (int k = 0;; ++k) {
            bool end = false;
            while (__builtin_amdgcn_readfirstlane(ctl[0]) <= k) { if (__builtin_amdgcn_readfirstlane(ctl[8]) <= k) { end = true; break; } __builtin_amdgcn_s_sleep(1); }
            if (end) break;
            if (__builtin_amdgcn_readfirstlane(ctl[0]) - k >= 3) __builtin_amdgcn_s_setprio(2); else __builtin_amdgcn_s_setprio(0);
            const LAS u8* rg = lds + CL_RING + (k % RING) * 4096 + lane * 16;
            u32x4 cur[4];
#pragma unroll
            for (int i = 0; i < 4; ++i) cur[i] = *(const LAS u32x4*)(rg + i * 1024);
            const int id = __builtin_amdgcn_readfirstlane(ctl[32 + (k % RING)]);
            WAIT_L(0);
            if (lane == 0) ctl[wid] = k + 1;
            const UnitDesc d = unit_desc(id);
#pragma unroll
            for (int i = 0; i < 4; ++i) __builtin_amdgcn_raw_buffer_store_b128(cur[i], rs, unit_dst(d, rank, i, lane), 0, RAUX);
            asm volatile("s_waitcnt vmcnt(%0)" :: "n"(4 * SWIN) : "memory");
            const int oldest = k >= SWIN - 1 ? hist[SWIN - 2 >= 0 ? SWIN - 2 : 0] : (k == 0 ? id : -1);
#pragma unroll
            for (int i = SWIN - 1; i > 0; --i) hist[i] = hist[i - 1];
            hist[0] = id;
            const int X = SWIN == 1 ? id : oldest;
            if (X >= 0 && lane == 0) __hip_atomic_store(prog, pbase + (unsigned)X, __ATOMIC_RELAXED, __HIP_MEMORY_SCOPE_SYSTEM);
        }
        __builtin_amdgcn_s_setprio(0);
        WAIT_V(0);
        if (lane == 0) __hip_atomic_store(prog, pbase + (unsigned)(PROG_MUL - 1), __ATOMIC_RELAXED, __HIP_MEMORY_SCOPE_SYSTEM);
    }
}

#define SA(b, h) (((b) * 2 + (h)) * HTB)
#define SB(b, h) ((4 + (b) * 2 + (h)) * HTB)
#define GLDS(gptr, ldsoff) do { const char* _g = (const char*)(gptr); asm volatile("" : "+s"(_g)); \
        __builtin_amdgcn_global_load_lds((const unsigned*)(_g + voff), (LAS unsigned*)(lds + (ldsoff) + ldsw), 16, 0, 0); } while (0)
#define STAGE(bufoff, gbase) do { GLDS((const char*)(gbase), (bufoff)); GLDS((const char*)(gbase) + 8192, (bufoff) + 8192); } while (0)
#define LDF(off) __builtin_shufflevector(*(const LAS i32x4*)(lds + (off)), *(const LAS i32x4*)(lds + (off) + 1024), 0, 1, 2, 3, 4, 5, 6, 7)
#define LDA(dst, b, h) do { _Pragma("unroll") for (int m = 0; m < 4; ++m) dst[m] = LDF(SA(b, h) + aoff + m * 2048); } while (0)
#define LDB(dst, b, h) do { _Pragma("unroll") for (int n = 0; n < 2; ++n) dst[n] = LDF(SB(b, h) + boff + n * 2048); } while (0)
#define C10(p) "a" #p "0","a" #p "1","a" #p "2","a" #p "3","a" #p "4","a" #p "5","a" #p "6","a" #p "7","a" #p "8","a" #p "9"
#define ACC_CLOB "a0","a1","a2","a3","a4","a5","a6","a7","a8","a9", C10(1),C10(2),C10(3),C10(4),C10(5),C10(6),C10(7),C10(8),C10(9),C10(10),C10(11),"a120","a121","a122","a123","a124","a125","a126","a127"
#define MF(o, A, B) "v_mfma_f32_16x16x128_f8f6f4 a[%c6+" #o ":%c6+" #o "+3], " A ", " B ", a[%c6+" #o ":%c6+" #o "+3]\n\t"
#define MFZ(o, A, B) "v_mfma_f32_16x16x128_f8f6f4 a[%c6+" #o ":%c6+" #o "+3], " A ", " B ", 0\n\t"
#define MMA(ai, bj, At, Bt) asm volatile("s_setprio 1\n\t" MF(0, "%0", "%4") MF(4, "%0", "%5") MF(8, "%1", "%4") MF(12, "%1", "%5") MF(16, "%2", "%4") MF(20, "%2", "%5") MF(24, "%3", "%4") MF(28, "%3", "%5") "s_setprio 0" \
        :: "v"(At[0]), "v"(At[1]), "v"(At[2]), "v"(At[3]), "v"(Bt[0]), "v"(Bt[1]), "i"(((ai) * 2 + (bj)) * 32) : ACC_CLOB)
#define MMAZ(ai, bj, At, Bt) asm volatile("s_setprio 1\n\t" MFZ(0, "%0", "%4") MFZ(4, "%0", "%5") MFZ(8, "%1", "%4") MFZ(12, "%1", "%5") MFZ(16, "%2", "%4") MFZ(20, "%2", "%5") MFZ(24, "%3", "%4") MFZ(28, "%3", "%5") "s_setprio 0" \
        :: "v"(At[0]), "v"(At[1]), "v"(At[2]), "v"(At[3]), "v"(Bt[0]), "v"(Bt[1]), "i"(((ai) * 2 + (bj)) * 32) : ACC_CLOB)
#define MM(Z, ai, bj, At, Bt) do { if (Z) MMAZ(ai, bj, At, Bt); else MMA(ai, bj, At, Bt); } while (0)
#define ACC_RD4(v, i0, i1, i2, i3) asm volatile("v_accvgpr_read_b32 %0, a[%c4]\n\tv_accvgpr_read_b32 %1, a[%c5]\n\tv_accvgpr_read_b32 %2, a[%c6]\n\tv_accvgpr_read_b32 %3, a[%c7]" \
        : "=v"((v)[0]), "=v"((v)[1]), "=v"((v)[2]), "=v"((v)[3]) : "i"(i0), "i"(i1), "i"(i2), "i"(i3))
#define EPI1(ai, m, r) do { float x_[4]; ACC_RD4(x_, (ai) * 64 + (m) * 8 + (r), (ai) * 64 + (m) * 8 + 4 + (r), (ai) * 64 + 32 + (m) * 8 + (r), (ai) * 64 + 32 + (m) * 8 + 4 + (r)); \
        f32x4 v = (f32x4){x_[0], x_[1], x_[2], x_[3]} * scl; \
        v[0] = v[0] < 0.f ? 0.f : v[0]; v[1] = v[1] < 0.f ? 0.f : v[1]; v[2] = v[2] < 0.f ? 0.f : v[2]; v[3] = v[3] < 0.f ? 0.f : v[3]; \
        __builtin_amdgcn_raw_buffer_store_b128(__builtin_bit_cast(u32x4, v), rso, eoff + (unsigned)(((ai) * 128 + (m) * 16 + (r)) * N * 4), 0, 0); } while (0)
#define EPIM(ai, m) do { EPI1(ai, m, 0); EPI1(ai, m, 1); EPI1(ai, m, 2); EPI1(ai, m, 3); } while (0)

__global__ __launch_bounds__(512, 2) void fused_k(Params p) {
    extern __shared__ __attribute__((aligned(16))) u8 lds_raw[];
    LAS u8* lds = (LAS u8*)lds_raw;
    const int tid = threadIdx.x, wid = __builtin_amdgcn_readfirstlane(tid >> 6);
    const int bx = blockIdx.x, rank = p.rank;
    u8* myws = p.ws[rank];
    const unsigned call = (unsigned)__builtin_amdgcn_readfirstlane(__hip_atomic_load((unsigned*)(myws + WS_CALL), __ATOMIC_RELAXED, __HIP_MEMORY_SCOPE_AGENT)) + 1u;
    unsigned* tmo = (unsigned*)(myws + WS_TMO);
    const int wr = wid >> 2, wc = wid & 3;
    const unsigned ldsw = (unsigned)wid * 1024u;
    volatile LAS int* lknown = (volatile LAS int*)(lds + LKNOWN_OFF);
    const size_t par = WS_DATA + (size_t)(call & 1) * PAR_BYTES;
    const char* Abase = (const char*)myws + par;
    const char* Bbase = Abase + A_BYTES;
    const unsigned* flags = (const unsigned*)(myws + WS_FLAGS);
    const float sc = p.sx[0] * p.sw[0];
    int known = 0;
    bool bad = false;
    auto ready = [&](int need) {
        if (known >= need) return;
        if (wid == 0) {
            const int lane = threadIdx.x & 63, c = lane & 31;
            const unsigned pbase = call * PROG_MUL;
            const unsigned long long t0 = __builtin_amdgcn_s_memrealtime();
            int kn;
            for (;;) {
                unsigned mn = 0x7fffffffu;
                unsigned v[4];
#pragma unroll
                for (int i = 0; i < 4; ++i) { const int src = (lane >> 5) + 2 * i;
                    v[i] = src == rank ? __hip_atomic_load((unsigned*)(myws + WS_LOCAL), __ATOMIC_RELAXED, __HIP_MEMORY_SCOPE_AGENT)
                                       : (c < C_WG ? __hip_atomic_load(flags + src * 32 + c, __ATOMIC_RELAXED, __HIP_MEMORY_SCOPE_SYSTEM) : 0xffffffffu); }
#pragma unroll
                for (int i = 0; i < 4; ++i) { const int src = (lane >> 5) + 2 * i; unsigned pre;
                    if (src == rank) pre = v[i] >= call * (unsigned)G_WG ? 0x7fffffffu : 0u;
                    else pre = c >= C_WG ? 0x7fffffffu : (v[i] >= pbase ? v[i] - pbase : 0u);
                    mn = pre < mn ? pre : mn; }
#pragma unroll
                for (int o = 32; o; o >>= 1) { const unsigned x = (unsigned)__shfl_xor((int)mn, o); mn = x < mn ? x : mn; }
                kn = __builtin_amdgcn_readfirstlane((int)mn);
                if (kn >= need) break;
                if ((unsigned)__builtin_amdgcn_readfirstlane(__hip_atomic_load(tmo, __ATOMIC_RELAXED, __HIP_MEMORY_SCOPE_AGENT)) == call || __builtin_amdgcn_s_memrealtime() - t0 > DEADLINE) {
                    if (lane == 0) __hip_atomic_store(tmo, call, __ATOMIC_RELAXED, __HIP_MEMORY_SCOPE_AGENT);
                    kn = 0x7fffffff; break; }
                { const unsigned long long w0 = __builtin_amdgcn_s_memrealtime(); const unsigned long long dt = (unsigned long long)((need - kn) * POLL_NUM / 16 + 30);
                  while (__builtin_amdgcn_s_memrealtime() - w0 < dt) __builtin_amdgcn_s_sleep(8); }
            }
            __builtin_amdgcn_fence(__ATOMIC_ACQUIRE, "");
            if (lane == 0) *lknown = kn;
            asm volatile("s_waitcnt vmcnt(0) lgkmcnt(0)" ::: "memory");
        }
        asm volatile("" ::: "memory"); BAR; asm volatile("" ::: "memory");
        known = __builtin_amdgcn_readfirstlane(*lknown);
        asm volatile("s_waitcnt lgkmcnt(0)" ::: "memory");
        if (known == 0x7fffffff) bad = true;
    };
    auto kt = [&](int n) { return (n & ~7) | ((rank + n) & 7); };
#define APTR(base, n) ((base) + (size_t)kt(n) * (NI * BLK))
#define BPTR(base, n) ((base) + (size_t)kt(n) * (NJ * BLK))
    i32x8 At[4], B0[2], B1[2];
    int sidx = 1; (void)sidx;

    STAMP(0);
    if (is_comm(bx)) {
        comm_role(p, call, bx, wid, lds);
        STAMP(1);
    } else {
        const int lane = tid & 63, fr = lane & 15, fq = lane >> 4;
        const unsigned voff = (unsigned)tid * 16u;
        const int aoff = lds_byte(wr * 64 + fr, fq * 16), boff = lds_byte(wc * 32 + fr, fq * 16);
        const int xcd = bx & 7, loc = (bx >> 3) - C_WG / 8;
        {
            const __amdgpu_buffer_rsrc_t rsl = __builtin_amdgcn_make_buffer_rsrc(myws + par, 0, (int)PAR_BYTES, 0x00020000);
            constexpr int NGW = G_WG * 8;
            static_assert(2 * NGW >= U_TOTAL, "prologue");
            const int gw = (bx - C_WG) * 8 + wid;
            const bool l1 = gw + NGW < U_TOTAL;
            LAS u8* tl = lds + wid * 8192;
            unit_load_lds(p, gw, tl, lane);
            if (l1) unit_load_lds(p, gw + NGW, tl + 4096, lane);
            WAIT_V(0);
            { const UnitDesc d0 = unit_desc(gw); u32x4 c0[4]; unit_transform(d0.isx, tl, lane, c0);
#pragma unroll
              for (int i = 0; i < 4; ++i) __builtin_amdgcn_raw_buffer_store_b128(c0[i], rsl, unit_dst(d0, rank, i, lane), 0, 16); }
            if (l1) { const UnitDesc d1 = unit_desc(gw + NGW); u32x4 c1[4]; unit_transform(d1.isx, tl + 4096, lane, c1);
#pragma unroll
                for (int i = 0; i < 4; ++i) __builtin_amdgcn_raw_buffer_store_b128(c1[i], rsl, unit_dst(d1, rank, i, lane), 0, 16); }
            asm volatile("s_waitcnt vmcnt(0) lgkmcnt(0)" ::: "memory");
            __syncthreads();
            if (tid == 0) __hip_atomic_fetch_add((unsigned*)(myws + WS_LOCAL), 1u, __ATOMIC_RELAXED, __HIP_MEMORY_SCOPE_AGENT);
            STAMP(20);
        }
        const int ti = 4 * (xcd & 3) + (loc & 3);
        int tj = 7 * (xcd >> 2) + (loc >> 2), gate = 7 * (xcd >> 2) + 6;
        const char* cA = Abase + (size_t)ti * BLK;
        const char* cB = Bbase + (size_t)tj * BLK;
        ready(uneed(gate, 0));
        STAMP(sidx); ++sidx;
        {
            const char* a0 = APTR(cA, 0); const char* b0 = BPTR(cB, 0); const char* a1 = APTR(cA, 1); const char* b1 = BPTR(cB, 1);
            STAGE(SB(0, 0), b0); STAGE(SB(0, 1), b0 + HTB); STAGE(SA(0, 0), a0); STAGE(SA(0, 1), a0 + HTB);
            STAGE(SB(1, 0), b1); STAGE(SB(1, 1), b1 + HTB); STAGE(SA(1, 0), a1);
            if (wr == 1) BAR;
            WAIT_V(6); BAR; BAR;
        }
        bool first = true; (void)first;
#define KT(b, MODE) do { \
                LDB(B0, b, 0); LDB(B1, b, 1); SCHED; LDA(At, b, 0); STAGE(SA((b) ^ 1, 1), an + HTB); \
                if (MODE == 1) { SCHED; EPIM(1, 0); EPIM(1, 1); EPIM(1, 2); EPIM(1, 3); SCHED; WAIT_V(40); } else if (MODE == 2) WAIT_V(24); else WAIT_V(8); \
                WAIT_L(0); BAR; MM(MODE == 1, 0, 0, At, B0); MM(MODE == 1, 0, 1, At, B1); BAR; SCHED; \
                LDA(At, b, 1); STAGE(SB(b, 0), b2); STAGE(SB(b, 1), b2 + HTB); STAGE(SA(b, 0), a2); \
                if (MODE == 3) { SCHED; asm volatile("s_nop 7"); EPIM(0, 0); EPIM(0, 1); EPIM(0, 2); EPIM(0, 3); SCHED; WAIT_V(24); } else if (MODE == 1) WAIT_V(40); else WAIT_V(8); \
                WAIT_L(0); BAR; MM(MODE == 1, 1, 0, At, B0); MM(MODE == 1, 1, 1, At, B1); BAR; SCHED; } while (0)
        const unsigned eoff0 = (unsigned)(((ti * 256 + wr * 64 + 4 * fq) * N + wc * 64 + 4 * fr) * 4);
        __amdgpu_buffer_rsrc_t rso = __builtin_amdgcn_make_buffer_rsrc(p.out, 0, 0, 0x00020000);
        unsigned eoff = eoff0;
        float scl = sc;
        for (;;) {
            const bool has_next = tj + HCOLS < 4 * HCOLS;
            const int ntj = tj + HCOLS, ngate = gate + HCOLS;
            const char* nB = has_next ? Bbase + (size_t)ntj * BLK : cB;
            {
                asm volatile("s_nop 7");
                { const char* an = APTR(cA, 1); const char* a2 = APTR(cA, 2); const char* b2 = BPTR(cB, 2); KT(0, 1); }
                { const char* an = APTR(cA, 2); const char* a2 = APTR(cA, 3); const char* b2 = BPTR(cB, 3); KT(1, 2); }
            }
            for (int t = 2; t < NT - 2; t += 2) {
                if (((t + 2) & 7) == 0) { if (first) { STAMP(sidx); ++sidx; } ready(uneed(gate, (t + 2) >> 3)); if (first) { STAMP(sidx); ++sidx; } }
                { const char* an = APTR(cA, t + 1); const char* a2 = APTR(cA, t + 2); const char* b2 = BPTR(cB, t + 2); KT(0, 0); }
                { const char* an = APTR(cA, t + 2); const char* a2 = APTR(cA, t + 3); const char* b2 = BPTR(cB, t + 3); KT(1, 0); }
            }
            {
                if (has_next) ready(uneed(ngate, 0));
                { const char* an = APTR(cA, NT - 1); const char* a2 = APTR(cA, 0); const char* b2 = BPTR(nB, 0); KT(0, 0); }
                rso = __builtin_amdgcn_make_buffer_rsrc(p.out, 0, M * N * 4, 0x00020000);
                eoff = eoff0 + (unsigned)tj * 1024u;
                scl = bad ? __builtin_nanf("") : sc;
                { const char* an = APTR(cA, 0); const char* a2 = APTR(cA, 1); const char* b2 = BPTR(nB, 1); KT(1, 3); }
            }
            STAMP(sidx); ++sidx;
            if (!has_next) break;
            gate = ngate; tj = ntj; cB = nB; first = false;
        }
        asm volatile("s_nop 15\n\ts_nop 15" ::: "memory");
        EPIM(1, 0); EPIM(1, 1); EPIM(1, 2); EPIM(1, 3);
        WAIT_V(0);
        if (wr == 0) BAR;
    }
    {
        asm volatile("s_waitcnt vmcnt(0) lgkmcnt(0)" ::: "memory");
        BAR;
        int tz = threadIdx.x; asm volatile("" : "+v"(tz));
        const int lane = tz & 63, fr = lane & 15, fq = lane >> 4;
        const unsigned voff = (unsigned)tz * 16u;
        const int aoff = lds_byte(wr * 64 + fr, fq * 16), boff = lds_byte(wc * 32 + fr, fq * 16);
        const int tj = 4 * HCOLS + 4 * ((bx >> 2) & 1) + ((bx >> 3) & 3), ti = 4 * (bx & 3) + ((bx >> 5) & 3), ah = bx >> 7;
        const char* cA = Abase + (size_t)ti * BLK + ah * HTB;
        const char* cB = Bbase + (size_t)tj * BLK;
        ready(uneed(tj, 3));
        constexpr int HB = 49152;
        { const char* a = APTR(cA, 0); const char* b = BPTR(cB, 0); STAGE(0, a); STAGE(HTB, b); STAGE(2 * HTB, b + HTB);
          a = APTR(cA, 1); b = BPTR(cB, 1); STAGE(HB, a); STAGE(HB + HTB, b); STAGE(HB + 2 * HTB, b + HTB); }
        if (wr == 1) BAR;
        WAIT_V(6); BAR; BAR;
        int bo = 0;
        for (int t = 0; t < NT; ++t) {
            int nbo = bo + 2 * HB; if (nbo >= 3 * HB) nbo -= 3 * HB;
            const bool more = t + 2 < NT;
            const char* a = APTR(cA, (more ? t + 2 : t)); const char* b = BPTR(cB, (more ? t + 2 : t));
#pragma unroll
            for (int n = 0; n < 2; ++n) B0[n] = LDF(bo + HTB + boff + n * 2048);
#pragma unroll
            for (int m = 0; m < 4; ++m) At[m] = LDF(bo + aoff + m * 2048);
#pragma unroll
            for (int n = 0; n < 2; ++n) B1[n] = LDF(bo + 2 * HTB + boff + n * 2048);
            if (more) { STAGE(nbo, a); STAGE(nbo + HTB, b); STAGE(nbo + 2 * HTB, b + HTB); WAIT_V(6); } else WAIT_V(0);
            WAIT_L(0); BAR; MM(t == 0, 0, 0, At, B0); MM(t == 0, 0, 1, At, B1); BAR; SCHED;
            bo += HB; if (bo >= 3 * HB) bo = 0;
        }
        if (wr == 0) BAR;
        asm volatile("s_nop 15\n\ts_nop 15" ::: "memory");
        const float scl = bad ? __builtin_nanf("") : sc;
        const __amdgpu_buffer_rsrc_t rso = __builtin_amdgcn_make_buffer_rsrc(p.out, 0, M * N * 4, 0x00020000);
        const unsigned eoff = (unsigned)(((ti * 256 + ah * 128 + wr * 64 + 4 * fq) * N + tj * 256 + wc * 64 + 4 * fr) * 4);
        EPIM(0, 0); EPIM(0, 1); EPIM(0, 2); EPIM(0, 3);
        STAMP(sidx); ++sidx;
    }
    __syncthreads();
    if (tid == 0) {
        const unsigned t = __hip_atomic_fetch_add((unsigned*)(myws + WS_DONE), 1u, __ATOMIC_RELAXED, __HIP_MEMORY_SCOPE_AGENT);
        if (t == call * GRID - 1u) __hip_atomic_store((unsigned*)(myws + WS_CALL), call, __ATOMIC_RELAXED, __HIP_MEMORY_SCOPE_AGENT);
    }
}
}

extern "C" void kernel_launch(int rank, int nranks, ncclComm_t comm, hipStream_t stream,
                              void* const* d_in, const int* in_sizes, int n_in,
                              void* d_out, int out_size,
                              const int* in_dtypes, int out_dtype,
                              void* const* d_out_all,
                              void* const* d_ws_all, size_t ws_bytes) {
    if (nranks != NR || rank < 0 || rank >= NR || ws_bytes < WS_NEED || n_in < 4 || out_size != M * N) { fprintf(stderr, "unexpected configuration\n"); return; }
    HIP_CHECK(hipFuncSetAttribute((const void*)fused_k, hipFuncAttributeMaxDynamicSharedMemorySize, LDS_BYTES));
    Params p;
    p.x = (const u8*)d_in[0]; p.w = (const u8*)d_in[1]; p.sx = (const float*)d_in[2]; p.sw = (const float*)d_in[3];
    p.out = (float*)d_out; p.rank = rank;
    for (int i = 0; i < NR; ++i) p.ws[i] = (u8*)d_ws_all[i];
    hipLaunchKernelGGL(fused_k, dim3(GRID), dim3(512), LDS_BYTES, stream, p);
    HIP_CHECK(hipGetLastError());
}
```

```cpp
#include <hip/hip_runtime.h>
#include <rccl/rccl.h>
#include <cstdio>

#define HIP_CHECK(expr) do { hipError_t _e = (expr); if (_e != hipSuccess) { \
    fprintf(stderr, "HIP error %s at %s:%d\n", hipGetErrorString(_e), __FILE__, __LINE__); \
    return; } } while (0)

namespace {
#define LAS __attribute__((address_space(3)))
typedef int i32x4 __attribute__((ext_vector_type(4)));
typedef int i32x8 __attribute__((ext_vector_type(8)));
typedef float f32x4 __attribute__((ext_vector_type(4)));
typedef unsigned u32x4 __attribute__((ext_vector_type(4)));
typedef unsigned char u8;

constexpr int M = 4096, N = 16384, K = 4096, NR = 8, KS = K / NR;
constexpr int HTB = 16384, BLK = 32768, STAGE_BYTES = 8 * HTB, LDS_BYTES = 163840, LKNOWN_OFF = LDS_BYTES - 64;
constexpr int NT = 32, NI = 16, NJ = 64;
#define CSTEP 16
#define RAUX 17
#define POLL_NUM 64
#define SWIN 1
constexpr int GRID = 256, C_WG = 32, G_WG = GRID - C_WG;
constexpr size_t WS_CALL = 0, WS_DONE = 128, WS_TMO = 256, WS_LOCAL = 384, WS_CLAIM = 512;
constexpr size_t WS_FLAGS = 65536;
constexpr size_t WS_DATA = 1 << 20;
constexpr size_t A_BYTES = (size_t)M * K, B_BYTES = (size_t)K * N, PAR_BYTES = A_BYTES + B_BYTES;
constexpr size_t WS_NEED = WS_DATA + 2 * PAR_BYTES;
constexpr unsigned long long DEADLINE = 50000000ull;
constexpr unsigned PROG_MUL = 4096;

struct Params { const u8* x; const u8* w; const float* sx; const float* sw; float* out; u8* ws[NR]; int rank; };

__device__ __forceinline__ int lds_byte(int r, int c) { const int st = (r >> 4) * 2 + (c >> 6), ob = (r & 15) * 64 + (c & 63); return st * 1024 + (ob ^ (((ob >> 9) & 1) << 5)); }
__host__ __device__ __forceinline__ bool is_comm(int bx) { return bx < C_WG; }

#define STAMP(idx) do {} while (0)
#define WAIT_V(n) asm volatile("s_waitcnt vmcnt(" #n ")" ::: "memory")
#define WAIT_L(n) asm volatile("s_waitcnt lgkmcnt(" #n ")" ::: "memory")
#define BAR __builtin_amdgcn_s_barrier()
#define SCHED __builtin_amdgcn_sched_barrier(0)

constexpr int HSTEP = 128 + 56, U_HEAD = 4 * HSTEP, RSTEP = 112, U_LAST = U_HEAD + 16 * RSTEP, U_TOTAL = U_LAST + 32;
static_assert(U_TOTAL == 2560 && G_WG == 224 && C_WG == 32, "schedule");
struct UnitDesc { int isx, kk, a, b; };
__device__ __forceinline__ UnitDesc unit_desc(int u) {
    UnitDesc d;
    if (u < U_HEAD) { const int kk = u / HSTEP, r = u - kk * HSTEP; d.kk = kk;
        if (r < 128) { d.isx = 1; d.a = r >> 3; d.b = r & 7; }
        else { const int q = r - 128; d.isx = 0; d.a = q >> 3; d.b = q & 7; }
    } else if (u < U_LAST) { const int v = u - U_HEAD, g = v / RSTEP, r = v - g * RSTEP; d.isx = 0; d.kk = g & 3; d.a = 7 + 14 * (g >> 2) + (r >> 3); d.b = r & 7; }
    else { const int v = u - U_LAST; d.isx = 0; d.kk = v >> 3; d.a = 63; d.b = v & 7; }
    return d;
}
__device__ __forceinline__ int uneed(int j, int kk) {
    if (j < 7) return kk * HSTEP + 128 + (j + 1) * 8;
    if (j < 63) { const int r = (j - 7) / 14; return U_HEAD + (r * 4 + kk) * RSTEP + (j - 7 - r * 14 + 1) * 8; }
    return U_LAST + (kk + 1) * 8;
}
__device__ __forceinline__ void unit_load_lds(const Params& p, int u, LAS u8* dst, int lane) {
    const UnitDesc d = unit_desc(u);
    const int s = lane >> 2, cqp = lane & 3;
    if (d.isx) {
        const int cq = cqp ^ ((s >> 3) << 1);
        const u8* src = p.x + (size_t)(d.a * 256 + (d.b >> 2) * 128 + (d.b & 3) * 32 + s) * KS + d.kk * 128 + cq * 16;
#pragma unroll
        for (int i = 0; i < 4; ++i) __builtin_amdgcn_global_load_lds((const unsigned*)(src + (size_t)((i >> 1) * 16) * KS + (i & 1) * 64), (LAS unsigned*)(dst + i * 1024), 16, 0, 0);
    } else {
        const int wc = d.b >> 1, kh = d.b & 1;
        const u8* src = p.w + (size_t)(d.kk * 128 + kh * 64 + s) * N + d.a * 256 + wc * 64 + cqp * 16;
#pragma unroll
        for (int i = 0; i < 4; ++i) __builtin_amdgcn_global_load_lds((const unsigned*)(src + (size_t)(i * 16) * N), (LAS unsigned*)(dst + i * 1024), 16, 0, 0);
    }
}
__device__ __forceinline__ void unit_transform(bool isx, const LAS u8* tl, int lane, u32x4 (&cur)[4]) {
    if (isx) {
#pragma unroll
        for (int i = 0; i < 4; ++i) cur[i] = *(const LAS u32x4*)(tl + i * 1024 + lane * 16);
    } else {
        const int s = lane >> 2, cqp = lane & 3;
        const int cq = cqp ^ ((s >> 3) << 1);
        const int rd_off = (cq * 16) * 64 + s * 4;
        unsigned ld[16];
#pragma unroll
        for (int i = 0; i < 16; ++i) ld[i] = *(const LAS unsigned*)(tl + rd_off + i * 64);
#pragma unroll
        for (int t = 0; t < 4; ++t) {
            const unsigned d0 = ld[4 * t], d1 = ld[4 * t + 1], d2 = ld[4 * t + 2], d3 = ld[4 * t + 3];
            const unsigned a = __builtin_amdgcn_perm(d1, d0, 0x05010400u), b = __builtin_amdgcn_perm(d1, d0, 0x07030602u);
            const unsigned c = __builtin_amdgcn_perm(d3, d2, 0x05010400u), e = __builtin_amdgcn_perm(d3, d2, 0x07030602u);
            cur[0][t] = __builtin_amdgcn_perm(c, a, 0x05040100u); cur[1][t] = __builtin_amdgcn_perm(c, a, 0x07060302u);
            cur[2][t] = __builtin_amdgcn_perm(e, b, 0x05040100u); cur[3][t] = __builtin_amdgcn_perm(e, b, 0x07060302u);
        }
    }
}
__device__ __forceinline__ unsigned unit_dst(const UnitDesc& d, int rank, int i, int lane) {
    if (d.isx) return (unsigned)(((d.kk * 8 + rank) * NI + d.a) * BLK + d.b * 4096 + i * 1024 + lane * 16);
    const int wc = d.b >> 1, kh = d.b & 1;
    return (unsigned)(A_BYTES + (size_t)((d.kk * 8 + rank) * NJ + d.a) * BLK + (i >> 1) * HTB + ((wc * 2 + (i & 1)) * 2 + kh) * 1024 + lane * 16);
}

#define RING 2
#define PRE 2
#define CLD 1
constexpr int CL_RING = 0, CL_RAW = RING * 4096, CL_CTL = CL_RAW + (PRE + 1) * 4096;
static_assert(CL_CTL + 256 <= LKNOWN_OFF, "comm LDS");
__device__ __forceinline__ void comm_role(const Params& p, unsigned call, int cwg, int wid, LAS u8* lds) {
    const int lane = threadIdx.x & 63;
    const int rank = p.rank;
    volatile LAS int* ctl = (volatile LAS int*)(lds + CL_CTL);
    if (threadIdx.x < 9) ctl[threadIdx.x] = threadIdx.x == 8 ? 0x7fffffff : 0;
    __syncthreads();
    if (wid == 0) {
        __builtin_amdgcn_s_setprio(3);
        unsigned* claim = (unsigned*)(p.ws[rank] + WS_CLAIM);
        const unsigned cbase = (call - 1u) * (unsigned)(U_TOTAL + C_WG * CLD);
        unsigned long long sexec;
        asm volatile("v_accvgpr_write_b32 a3, 1\n\ts_nop 1" ::: "a3");
#define CLAIM(reg) asm volatile("s_mov_b64 %0, exec\n\ts_mov_b64 exec, 1\n\tglobal_atomic_add a" #reg ", %1, a3, off sc0\n\ts_mov_b64 exec, %0" : "=&s"(sexec) : "v"(claim) : "memory", "a0", "a1", "a2", "a3")
        CLAIM(0); if (CLD > 1) CLAIM(1); if (CLD > 2) CLAIM(2);
        int j = 0, nl = 0x7fffffff;
        int lslot = 0, pslot = 0;
        bool fin = false;
#define STEP(reg) do { \
            if (nl == 0x7fffffff) { \
                if (j < CLD) WAIT_V(0); else asm volatile("s_waitcnt vmcnt(%0)" :: "n"(5 * (CLD - 1)) : "memory"); \
                unsigned cv; asm volatile("v_accvgpr_read_b32 %0, a" #reg : "=v"(cv)); \
                const int id = (int)((unsigned)__builtin_amdgcn_readfirstlane(cv) - cbase); \
                if (id < U_TOTAL) { unit_load_lds(p, id, lds + CL_RAW + lslot * 4096, lane); if (lane == 0) ctl[16 + lslot] = id; CLAIM(reg); if (++lslot > PRE) lslot = 0; } \
                else { nl = j; if (lane == 0) ctl[8] = nl; } \
            } \
            const int k = j - PRE; \
            if (k >= 0) { \
                if (k >= nl) { fin = true; } else { \
                    if (nl == 0x7fffffff) asm volatile("s_waitcnt vmcnt(%0)" :: "n"(5 * PRE + 1) : "memory"); else WAIT_V(0); \
                    const int id = __builtin_amdgcn_readfirstlane(ctl[16 + pslot]); \
                    u32x4 cur[4]; \
                    unit_transform(unit_desc(id).isx, lds + CL_RAW + pslot * 4096, lane, cur); \
                    if (k >= RING) { for (;;) { int mn = ctl[1 + (lane % 7)]; \
                            _Pragma("unroll") for (int o = 4; o; o >>= 1) { const int x = __shfl_xor(mn, o); mn = x < mn ? x : mn; } \
                            if (__builtin_amdgcn_readfirstlane(mn) > k - RING) break; \
                            __builtin_amdgcn_s_sleep(1); } } \
                    LAS u8* rg = lds + CL_RING + (k % RING) * 4096 + lane * 16; \
                    _Pragma("unroll") for (int i = 0; i < 4; ++i) *(LAS u32x4*)(rg + i * 1024) = cur[i]; \
                    if (lane == 0) ctl[32 + (k % RING)] = id; \
                    WAIT_L(0); \
                    if (lane == 0) ctl[0] = k + 1; \
                    if (++pslot > PRE) pslot = 0; \
                } } \
            ++j; } while (0)
        while (!fin) { STEP(0); if (fin) break; if (CLD > 1) { STEP(1); if (fin) break; } if (CLD > 2) STEP(2); }
        WAIT_V(0);
        __builtin_amdgcn_s_setprio(0);
    } else {
        const int dest = (rank + wid) & 7;
        u8* dws = p.ws[dest];
        const __amdgpu_buffer_rsrc_t rs = __builtin_amdgcn_make_buffer_rsrc(dws + WS_DATA + (size_t)(call & 1) * PAR_BYTES, 0, (int)PAR_BYTES, 0x00020000);
        unsigned* prog = (unsigned*)(dws + WS_FLAGS) + rank * 32 + cwg;
        const unsigned pbase = call * PROG_MUL;
        int hist[SWIN];
#pragma unroll
        for (int i = 0; i < SWIN; ++i) hist[i] = -1;
        for (int k = 0;; ++k) {
            bool end = false;
            while (__builtin_amdgcn_readfirstlane(ctl[0]) <= k) { if (__builtin_amdgcn_readfirstlane(ctl[8]) <= k) { end = true; break; } __builtin_amdgcn_s_sleep(1); }
            if (end) break;
            if (__builtin_amdgcn_readfirstlane(ctl[0]) - k >= 3) __builtin_amdgcn_s_setprio(2); else __builtin_amdgcn_s_setprio(0);
            const LAS u8* rg = lds + CL_RING + (k % RING) * 4096 + lane * 16;
            u32x4 cur[4];
#pragma unroll
            for (int i = 0; i < 4; ++i) cur[i] = *(const LAS u32x4*)(rg + i * 1024);
            const int id = __builtin_amdgcn_readfirstlane(ctl[32 + (k % RING)]);
            WAIT_L(0);
            if (lane == 0) ctl[wid] = k + 1;
            const UnitDesc d = unit_desc(id);
#pragma unroll
            for (int i = 0; i < 4; ++i) __builtin_amdgcn_raw_buffer_store_b128(cur[i], rs, unit_dst(d, rank, i, lane), 0, RAUX);
            asm volatile("s_waitcnt vmcnt(%0)" :: "n"(4 * SWIN) : "memory");
            const int oldest = k >= SWIN - 1 ? hist[SWIN - 2 >= 0 ? SWIN - 2 : 0] : (k == 0 ? id : -1);
#pragma unroll
            for (int i = SWIN - 1; i > 0; --i) hist[i] = hist[i - 1];
            hist[0] = id;
            const int X = SWIN == 1 ? id : oldest;
            if (X >= 0 && lane == 0) __hip_atomic_store(prog, pbase + (unsigned)X, __ATOMIC_RELAXED, __HIP_MEMORY_SCOPE_SYSTEM);
        }
        __builtin_amdgcn_s_setprio(0);
        WAIT_V(0);
        if (lane == 0) __hip_atomic_store(prog, pbase + (unsigned)(PROG_MUL - 1), __ATOMIC_RELAXED, __HIP_MEMORY_SCOPE_SYSTEM);
    }
}

#define SA(b, h) (((b) * 2 + (h)) * HTB)
#define SB(b, h) ((4 + (b) * 2 + (h)) * HTB)
#define GLDS(gptr, ldsoff) do { const char* _g = (const char*)(gptr); asm volatile("" : "+s"(_g)); \
        __builtin_amdgcn_global_load_lds((const unsigned*)(_g + voff), (LAS unsigned*)(lds + (ldsoff) + ldsw), 16, 0, 0); } while (0)
#define STAGE(bufoff, gbase) do { GLDS((const char*)(gbase), (bufoff)); GLDS((const char*)(gbase) + 8192, (bufoff) + 8192); } while (0)
#define LDF(off) __builtin_shufflevector(*(const LAS i32x4*)(lds + (off)), *(const LAS i32x4*)(lds + (off) + 1024), 0, 1, 2, 3, 4, 5, 6, 7)
#define LDA(dst, b, h) do { _Pragma("unroll") for (int m = 0; m < 4; ++m) dst[m] = LDF(SA(b, h) + aoff + m * 2048); } while (0)
#define LDB(dst, b, h) do { _Pragma("unroll") for (int n = 0; n < 2; ++n) dst[n] = LDF(SB(b, h) + boff + n * 2048); } while (0)
#define C10(p) "a" #p "0","a" #p "1","a" #p "2","a" #p "3","a" #p "4","a" #p "5","a" #p "6","a" #p "7","a" #p "8","a" #p "9"
#define ACC_CLOB "a0","a1","a2","a3","a4","a5","a6","a7","a8","a9", C10(1),C10(2),C10(3),C10(4),C10(5),C10(6),C10(7),C10(8),C10(9),C10(10),C10(11),"a120","a121","a122","a123","a124","a125","a126","a127"
#define MF(o, A, B) "v_mfma_f32_16x16x128_f8f6f4 a[%c6+" #o ":%c6+" #o "+3], " A ", " B ", a[%c6+" #o ":%c6+" #o "+3]\n\t"
#define MFZ(o, A, B) "v_mfma_f32_16x16x128_f8f6f4 a[%c6+" #o ":%c6+" #o "+3], " A ", " B ", 0\n\t"
#define MMA(ai, bj, At, Bt) asm volatile("s_setprio 1\n\t" MF(0, "%0", "%4") MF(4, "%0", "%5") MF(8, "%1", "%4") MF(12, "%1", "%5") MF(16, "%2", "%4") MF(20, "%2", "%5") MF(24, "%3", "%4") MF(28, "%3", "%5") "s_setprio 0" \
        :: "v"(At[0]), "v"(At[1]), "v"(At[2]), "v"(At[3]), "v"(Bt[0]), "v"(Bt[1]), "i"(((ai) * 2 + (bj)) * 32) : ACC_CLOB)
#define MMAZ(ai, bj, At, Bt) asm volatile("s_setprio 1\n\t" MFZ(0, "%0", "%4") MFZ(4, "%0", "%5") MFZ(8, "%1", "%4") MFZ(12, "%1", "%5") MFZ(16, "%2", "%4") MFZ(20, "%2", "%5") MFZ(24, "%3", "%4") MFZ(28, "%3", "%5") "s_setprio 0" \
        :: "v"(At[0]), "v"(At[1]), "v"(At[2]), "v"(At[3]), "v"(Bt[0]), "v"(Bt[1]), "i"(((ai) * 2 + (bj)) * 32) : ACC_CLOB)
#define MM(Z, ai, bj, At, Bt) do { if (Z) MMAZ(ai, bj, At, Bt); else MMA(ai, bj, At, Bt); } while (0)
#define ACC_RD4(v, i0, i1, i2, i3) asm volatile("v_accvgpr_read_b32 %0, a[%c4]\n\tv_accvgpr_read_b32 %1, a[%c5]\n\tv_accvgpr_read_b32 %2, a[%c6]\n\tv_accvgpr_read_b32 %3, a[%c7]" \
        : "=v"((v)[0]), "=v"((v)[1]), "=v"((v)[2]), "=v"((v)[3]) : "i"(i0), "i"(i1), "i"(i2), "i"(i3))
#define EPI1(ai, m, r) do { float x_[4]; ACC_RD4(x_, (ai) * 64 + (m) * 8 + (r), (ai) * 64 + (m) * 8 + 4 + (r), (ai) * 64 + 32 + (m) * 8 + (r), (ai) * 64 + 32 + (m) * 8 + 4 + (r)); \
        f32x4 v = (f32x4){x_[0], x_[1], x_[2], x_[3]} * scl; \
        v[0] = v[0] < 0.f ? 0.f : v[0]; v[1] = v[1] < 0.f ? 0.f : v[1]; v[2] = v[2] < 0.f ? 0.f : v[2]; v[3] = v[3] < 0.f ? 0.f : v[3]; \
        __builtin_amdgcn_raw_buffer_store_b128(__builtin_bit_cast(u32x4, v), rso, eoff + (unsigned)(((ai) * 128 + (m) * 16 + (r)) * N * 4), 0, 0); } while (0)
#define EPIM(ai, m) do { EPI1(ai, m, 0); EPI1(ai, m, 1); EPI1(ai, m, 2); EPI1(ai, m, 3); } while (0)

__global__ __launch_bounds__(512, 2) void fused_k(Params p) {
    extern __shared__ __attribute__((aligned(16))) u8 lds_raw[];
    LAS u8* lds = (LAS u8*)lds_raw;
    const int tid = threadIdx.x, wid = __builtin_amdgcn_readfirstlane(tid >> 6);
    const int bx = blockIdx.x, rank = p.rank;
    u8* myws = p.ws[rank];
    const unsigned call = (unsigned)__builtin_amdgcn_readfirstlane(__hip_atomic_load((unsigned*)(myws + WS_CALL), __ATOMIC_RELAXED, __HIP_MEMORY_SCOPE_AGENT)) + 1u;
    unsigned* tmo = (unsigned*)(myws + WS_TMO);
    const int wr = wid >> 2, wc = wid & 3;
    const unsigned ldsw = (unsigned)wid * 1024u;
    volatile LAS int* lknown = (volatile LAS int*)(lds + LKNOWN_OFF);
    const size_t par = WS_DATA + (size_t)(call & 1) * PAR_BYTES;
    const char* Abase = (const char*)myws + par;
    const char* Bbase = Abase + A_BYTES;
    const unsigned* flags = (const unsigned*)(myws + WS_FLAGS);
    const float sc = p.sx[0] * p.sw[0];
    int known = 0;
    bool bad = false;
    auto ready = [&](int need) {
        if (known >= need) return;
        if (wid == 0) {
            const int lane = threadIdx.x & 63, c = lane & 31;
            const unsigned pbase = call * PROG_MUL;
            const unsigned long long t0 = __builtin_amdgcn_s_memrealtime();
            int kn;
            for (;;) {
                unsigned mn = 0x7fffffffu;
                unsigned v[4];
#pragma unroll
                for (int i = 0; i < 4; ++i) { const int src = (lane >> 5) + 2 * i;
                    v[i] = src == rank ? __hip_atomic_load((unsigned*)(myws + WS_LOCAL), __ATOMIC_RELAXED, __HIP_MEMORY_SCOPE_AGENT)
                                       : (c < C_WG ? __hip_atomic_load(flags + src * 32 + c, __ATOMIC_RELAXED, __HIP_MEMORY_SCOPE_SYSTEM) : 0xffffffffu); }
#pragma unroll
                for (int i = 0; i < 4; ++i) { const int src = (lane >> 5) + 2 * i; unsigned pre;
                    if (src == rank) pre = v[i] >= call * (unsigned)G_WG ? 0x7fffffffu : 0u;
                    else pre = c >= C_WG ? 0x7fffffffu : (v[i] >= pbase ? v[i] - pbase : 0u);
                    mn = pre < mn ? pre : mn; }
#pragma unroll
                for (int o = 32; o; o >>= 1) { const unsigned x = (unsigned)__shfl_xor((int)mn, o); mn = x < mn ? x : mn; }
                kn = __builtin_amdgcn_readfirstlane((int)mn);
                if (kn >= need) break;
                if ((unsigned)__builtin_amdgcn_readfirstlane(__hip_atomic_load(tmo, __ATOMIC_RELAXED, __HIP_MEMORY_SCOPE_AGENT)) == call || __builtin_amdgcn_s_memrealtime() - t0 > DEADLINE) {
                    if (lane == 0) __hip_atomic_store(tmo, call, __ATOMIC_RELAXED, __HIP_MEMORY_SCOPE_AGENT);
                    kn = 0x7fffffff; break; }
                { const unsigned long long w0 = __builtin_amdgcn_s_memrealtime(); const unsigned long long dt = (unsigned long long)((need - kn) * POLL_NUM / 16 + 30);
                  while (__builtin_amdgcn_s_memrealtime() - w0 < dt) __builtin_amdgcn_s_sleep(8); }
            }
            __builtin_amdgcn_fence(__ATOMIC_ACQUIRE, "");
            if (lane == 0) *lknown = kn;
            asm volatile("s_waitcnt vmcnt(0) lgkmcnt(0)" ::: "memory");
        }
        asm volatile("" ::: "memory"); BAR; asm volatile("" ::: "memory");
        known = __builtin_amdgcn_readfirstlane(*lknown);
        asm volatile("s_waitcnt lgkmcnt(0)" ::: "memory");
        if (known == 0x7fffffff) bad = true;
    };
    auto kt = [&](int n) { return (n & ~7) | ((rank + n) & 7); };
#define APTR(base, n) ((base) + (size_t)kt(n) * (NI * BLK))
#define BPTR(base, n) ((base) + (size_t)kt(n) * (NJ * BLK))
    i32x8 At[4], B0[2], B1[2];
    int sidx = 1; (void)sidx;

    STAMP(0);
    if (is_comm(bx)) {
        comm_role(p, call, bx, wid, lds);
        STAMP(1);
    } else {
        const int lane = tid & 63, fr = lane & 15, fq = lane >> 4;
        const unsigned voff = (unsigned)tid * 16u;
        const int aoff = lds_byte(wr * 64 + fr, fq * 16), boff = lds_byte(wc * 32 + fr, fq * 16);
        const int xcd = bx & 7, loc = (bx >> 3) - C_WG / 8;
        {
            const __amdgpu_buffer_rsrc_t rsl = __builtin_amdgcn_make_buffer_rsrc(myws + par, 0, (int)PAR_BYTES, 0x00020000);
            constexpr int NGW = G_WG * 8;
            static_assert(2 * NGW >= U_TOTAL, "prologue");
            const int gw = (bx - C_WG) * 8 + wid;
            const bool l1 = gw + NGW < U_TOTAL;
            LAS u8* tl = lds + wid * 8192;
            unit_load_lds(p, gw, tl, lane);
            if (l1) unit_load_lds(p, gw + NGW, tl + 4096, lane);
            WAIT_V(0);
            { const UnitDesc d0 = unit_desc(gw); u32x4 c0[4]; unit_transform(d0.isx, tl, lane, c0);
#pragma unroll
              for (int i = 0; i < 4; ++i) __builtin_amdgcn_raw_buffer_store_b128(c0[i], rsl, unit_dst(d0, rank, i, lane), 0, 16); }
            if (l1) { const UnitDesc d1 = unit_desc(gw + NGW); u32x4 c1[4]; unit_transform(d1.isx, tl + 4096, lane, c1);
#pragma unroll
                for (int i = 0; i < 4; ++i) __builtin_amdgcn_raw_buffer_store_b128(c1[i], rsl, unit_dst(d1, rank, i, lane), 0, 16); }
            asm volatile("s_waitcnt vmcnt(0) lgkmcnt(0)" ::: "memory");
            __syncthreads();
            if (tid == 0) __hip_atomic_fetch_add((unsigned*)(myws + WS_LOCAL), 1u, __ATOMIC_RELAXED, __HIP_MEMORY_SCOPE_AGENT);
            STAMP(30);
        }
    }
    {
        asm volatile("s_waitcnt vmcnt(0) lgkmcnt(0)" ::: "memory");
        BAR;
        int tz = threadIdx.x; asm volatile("" : "+v"(tz));
        const int lane = tz & 63, fr = lane & 15, fq = lane >> 4;
        const unsigned voff = (unsigned)tz * 16u;
        const int aoff = lds_byte(wr * 64 + fr, fq * 16), boff = lds_byte(wc * 32 + fr, fq * 16);
        int ti, tj, ah;
        if (is_comm(bx)) { ti = bx >> 1; ah = bx & 1; tj = 63; }
        else { const int loc = (bx >> 3) - C_WG / 8; ti = 4 * (bx & 3) + (loc & 3); tj = loc >> 2; ah = (bx >> 2) & 1; }
        const int gate = is_comm(bx) ? 63 : 6;
        const char* cA = Abase + (size_t)ti * BLK + ah * HTB;
        const char* cB = Bbase + (size_t)tj * BLK;
        ready(uneed(gate, 0));
        STAMP(sidx); ++sidx;
        constexpr int HB = 49152;
        { const char* a = APTR(cA, 0); const char* b = BPTR(cB, 0); STAGE(0, a); STAGE(HTB, b); STAGE(2 * HTB, b + HTB);
          a = APTR(cA, 1); b = BPTR(cB, 1); STAGE(HB, a); STAGE(HB + HTB, b); STAGE(HB + 2 * HTB, b + HTB); }
        if (wr == 1) BAR;
        WAIT_V(6); BAR; BAR;
        int bo = 0;
        for (int t = 0; t < NT; ++t) {
            int nbo = bo + 2 * HB; if (nbo >= 3 * HB) nbo -= 3 * HB;
            const bool more = t + 2 < NT;
            if (more && ((t + 2) & 7) == 0) { STAMP(sidx); ++sidx; ready(uneed(gate, (t + 2) >> 3)); STAMP(sidx); ++sidx; }
            const char* a = APTR(cA, (more ? t + 2 : t)); const char* b = BPTR(cB, (more ? t + 2 : t));
#pragma unroll
            for (int n = 0; n < 2; ++n) B0[n] = LDF(bo + HTB + boff + n * 2048);
#pragma unroll
            for (int m = 0; m < 4; ++m) At[m] = LDF(bo + aoff + m * 2048);
#pragma unroll
            for (int n = 0; n < 2; ++n) B1[n] = LDF(bo + 2 * HTB + boff + n * 2048);
            if (more) { STAGE(nbo, a); STAGE(nbo + HTB, b); STAGE(nbo + 2 * HTB, b + HTB); WAIT_V(6); } else WAIT_V(0);
            WAIT_L(0); BAR; MM(t == 0, 0, 0, At, B0); MM(t == 0, 0, 1, At, B1); BAR; SCHED;
            bo += HB; if (bo >= 3 * HB) bo = 0;
        }
        if (wr == 0) BAR;
        asm volatile("s_nop 15\n\ts_nop 15" ::: "memory");
        const float scl = bad ? __builtin_nanf("") : sc;
        const __amdgpu_buffer_rsrc_t rso = __builtin_amdgcn_make_buffer_rsrc(p.out, 0, M * N * 4, 0x00020000);
        const unsigned eoff = (unsigned)(((ti * 256 + ah * 128 + wr * 64 + 4 * fq) * N + tj * 256 + wc * 64 + 4 * fr) * 4);
        EPIM(0, 0); EPIM(0, 1); EPIM(0, 2); EPIM(0, 3);
        STAMP(sidx); ++sidx;
    }
    if (!is_comm(bx)) {
        int tz = threadIdx.x; asm volatile("" : "+v"(tz));
        const int lane = tz & 63, fr = lane & 15, fq = lane >> 4;
        const unsigned voff = (unsigned)tz * 16u;
        const int aoff = lds_byte(wr * 64 + fr, fq * 16), boff = lds_byte(wc * 32 + fr, fq * 16);
        const int xcd = bx & 7, loc = (bx >> 3) - C_WG / 8;
        const int ti = 4 * (xcd & 3) + (loc & 3);
        int tj = 7 + 7 * (xcd >> 2) + (loc >> 2), gate = 7 + 7 * (xcd >> 2) + 6;
        const char* cA = Abase + (size_t)ti * BLK;
        const char* cB = Bbase + (size_t)tj * BLK;
        ready(uneed(gate, 0));
        STAMP(sidx); ++sidx;
        {
            const char* a0 = APTR(cA, 0); const char* b0 = BPTR(cB, 0); const char* a1 = APTR(cA, 1); const char* b1 = BPTR(cB, 1);
            STAGE(SB(0, 0), b0); STAGE(SB(0, 1), b0 + HTB); STAGE(SA(0, 0), a0); STAGE(SA(0, 1), a0 + HTB);
            STAGE(SB(1, 0), b1); STAGE(SB(1, 1), b1 + HTB); STAGE(SA(1, 0), a1);
            if (wr == 1) BAR;
            WAIT_V(6); BAR; BAR;
        }
        bool first = true; (void)first;
#define KT(b, MODE) do { \
                LDB(B0, b, 0); LDB(B1, b, 1); SCHED; LDA(At, b, 0); STAGE(SA((b) ^ 1, 1), an + HTB); \
                if (MODE == 1) { SCHED; EPIM(1, 0); EPIM(1, 1); EPIM(1, 2); EPIM(1, 3); SCHED; WAIT_V(40); } else if (MODE == 2) WAIT_V(24); else WAIT_V(8); \
                WAIT_L(0); BAR; MM(MODE == 1, 0, 0, At, B0); MM(MODE == 1, 0, 1, At, B1); BAR; SCHED; \
                LDA(At, b, 1); STAGE(SB(b, 0), b2); STAGE(SB(b, 1), b2 + HTB); STAGE(SA(b, 0), a2); \
                if (MODE == 3) { SCHED; asm volatile("s_nop 7"); EPIM(0, 0); EPIM(0, 1); EPIM(0, 2); EPIM(0, 3); SCHED; WAIT_V(24); } else if (MODE == 1) WAIT_V(40); else WAIT_V(8); \
                WAIT_L(0); BAR; MM(MODE == 1, 1, 0, At, B0); MM(MODE == 1, 1, 1, At, B1); BAR; SCHED; } while (0)
        const unsigned eoff0 = (unsigned)(((ti * 256 + wr * 64 + 4 * fq) * N + wc * 64 + 4 * fr) * 4);
        __amdgpu_buffer_rsrc_t rso = __builtin_amdgcn_make_buffer_rsrc(p.out, 0, 0, 0x00020000);
        unsigned eoff = eoff0;
        float scl = sc;
        for (;;) {
            const bool has_next = tj + 14 < 63;
            const int ntj = tj + 14, ngate = gate + 14;
            const char* nB = has_next ? Bbase + (size_t)ntj * BLK : cB;
            {
                asm volatile("s_nop 7");
                { const char* an = APTR(cA, 1); const char* a2 = APTR(cA, 2); const char* b2 = BPTR(cB, 2); KT(0, 1); }
                { const char* an = APTR(cA, 2); const char* a2 = APTR(cA, 3); const char* b2 = BPTR(cB, 3); KT(1, 2); }
            }
            for (int t = 2; t < NT - 2; t += 2) {
                if (((t + 2) & 7) == 0) { if (first) { STAMP(sidx); ++sidx; } ready(uneed(gate, (t + 2) >> 3)); if (first) { STAMP(sidx); ++sidx; } }
                { const char* an = APTR(cA, t + 1); const char* a2 = APTR(cA, t + 2); const char* b2 = BPTR(cB, t + 2); KT(0, 0); }
                { const char* an = APTR(cA, t + 2); const char* a2 = APTR(cA, t + 3); const char* b2 = BPTR(cB, t + 3); KT(1, 0); }
            }
            {
                if (has_next) ready(uneed(ngate, 0));
                { const char* an = APTR(cA, NT - 1); const char* a2 = APTR(cA, 0); const char* b2 = BPTR(nB, 0); KT(0, 0); }
                rso = __builtin_amdgcn_make_buffer_rsrc(p.out, 0, M * N * 4, 0x00020000);
                eoff = eoff0 + (unsigned)tj * 1024u;
                scl = bad ? __builtin_nanf("") : sc;
                { const char* an = APTR(cA, 0); const char* a2 = APTR(cA, 1); const char* b2 = BPTR(nB, 1); KT(1, 3); }
            }
            STAMP(sidx); ++sidx;
            if (!has_next) break;
            gate = ngate; tj = ntj; cB = nB; first = false;
        }
        asm volatile("s_nop 15\n\ts_nop 15" ::: "memory");
        EPIM(1, 0); EPIM(1, 1); EPIM(1, 2); EPIM(1, 3);
        WAIT_V(0);
        if (wr == 0) BAR;
    }
    __syncthreads();
    if (tid == 0) {
        const unsigned t = __hip_atomic_fetch_add((unsigned*)(myws + WS_DONE), 1u, __ATOMIC_RELAXED, __HIP_MEMORY_SCOPE_AGENT);
        if (t == call * GRID - 1u) __hip_atomic_store((unsigned*)(myws + WS_CALL), call, __ATOMIC_RELAXED, __HIP_MEMORY_SCOPE_AGENT);
    }
}
}

extern "C" void kernel_launch(int rank, int nranks, ncclComm_t comm, hipStream_t stream,
                              void* const* d_in, const int* in_sizes, int n_in,
                              void* d_out, int out_size,
                              const int* in_dtypes, int out_dtype,
                              void* const* d_out_all,
                              void* const* d_ws_all, size_t ws_bytes) {
    if (nranks != NR || rank < 0 || rank >= NR || ws_bytes < WS_NEED || n_in < 4 || out_size != M * N) { fprintf(stderr, "unexpected configuration\n"); return; }
    HIP_CHECK(hipFuncSetAttribute((const void*)fused_k, hipFuncAttributeMaxDynamicSharedMemorySize, LDS_BYTES));
    Params p;
    p.x = (const u8*)d_in[0]; p.w = (const u8*)d_in[1]; p.sx = (const float*)d_in[2]; p.sw = (const float*)d_in[3];
    p.out = (float*)d_out; p.rank = rank;
    for (int i = 0; i < NR; ++i) p.ws[i] = (u8*)d_ws_all[i];
    hipLaunchKernelGGL(fused_k, dim3(GRID), dim3(512), LDS_BYTES, stream, p);
    HIP_CHECK(hipGetLastError());
}
```

```cpp
#include <hip/hip_runtime.h>
#include <rccl/rccl.h>
#include <cstdio>

#define HIP_CHECK(expr) do { hipError_t _e = (expr); if (_e != hipSuccess) { \
    fprintf(stderr, "HIP error %s at %s:%d\n", hipGetErrorString(_e), __FILE__, __LINE__); \
    return; } } while (0)

namespace {
#define LAS __attribute__((address_space(3)))
typedef int i32x4 __attribute__((ext_vector_type(4)));
typedef int i32x8 __attribute__((ext_vector_type(8)));
typedef float f32x4 __attribute__((ext_vector_type(4)));
typedef unsigned u32x4 __attribute__((ext_vector_type(4)));
typedef unsigned char u8;

constexpr int M = 4096, N = 16384, K = 4096, NR = 8, KS = K / NR;
constexpr int HTB = 16384, BLK = 32768, STAGE_BYTES = 8 * HTB, LDS_BYTES = 163840, LKNOWN_OFF = LDS_BYTES - 64;
constexpr int NT = 32, NI = 16, NJ = 64;
#define CSTEP 16
#define RAUX 17
#define POLL_NUM 64
#define SWIN 1
constexpr int GRID = 256, C_WG = 32, G_WG = GRID - C_WG;
constexpr size_t WS_CALL = 0, WS_DONE = 128, WS_TMO = 256, WS_LOCAL = 384, WS_CLAIM = 512;
constexpr size_t WS_FLAGS = 65536;
constexpr size_t WS_DATA = 1 << 20;
constexpr size_t A_BYTES = (size_t)M * K, B_BYTES = (size_t)K * N, PAR_BYTES = A_BYTES + B_BYTES;
constexpr size_t WS_NEED = WS_DATA + 2 * PAR_BYTES;
constexpr unsigned long long DEADLINE = 50000000ull;
constexpr unsigned PROG_MUL = 4096;

struct Params { const u8* x; const u8* w; const float* sx; const float* sw; float* out; u8* ws[NR]; int rank; };

__device__ __forceinline__ int lds_byte(int r, int c) { const int st = (r >> 4) * 2 + (c >> 6), ob = (r & 15) * 64 + (c & 63); return st * 1024 + (ob ^ (((ob >> 9) & 1) << 5)); }
__host__ __device__ __forceinline__ bool is_comm(int bx) { return bx < C_WG; }

#define STAMP(idx) do {} while (0)
#define WAIT_V(n) asm volatile("s_waitcnt vmcnt(" #n ")" ::: "memory")
#define WAIT_L(n) asm volatile("s_waitcnt lgkmcnt(" #n ")" ::: "memory")
#define BAR __builtin_amdgcn_s_barrier()
#define SCHED __builtin_amdgcn_sched_barrier(0)

constexpr int HSTEP = 128 + 56, U_HEAD = 4 * HSTEP, RSTEP = 112, U_LAST = U_HEAD + 16 * RSTEP, U_TOTAL = U_LAST + 32;
static_assert(U_TOTAL == 2560 && G_WG == 224 && C_WG == 32, "schedule");
struct UnitDesc { int isx, kk, a, b; };
__device__ __forceinline__ UnitDesc unit_desc(int u) {
    UnitDesc d;
    if (u < U_HEAD) { const int kk = u / HSTEP, r = u - kk * HSTEP; d.kk = kk;
        if (r < 128) { d.isx = 1; d.a = r >> 3; d.b = r & 7; }
        else { const int q = r - 128; d.isx = 0; d.a = q >> 3; d.b = q & 7; }
    } else if (u < U_LAST) { const int v = u - U_HEAD, g = v / RSTEP, r = v - g * RSTEP; d.isx = 0; d.kk = g & 3; d.a = 7 + 14 * (g >> 2) + (r >> 3); d.b = r & 7; }
    else { const int v = u - U_LAST; d.isx = 0; d.kk = v >> 3; d.a = 63; d.b = v & 7; }
    return d;
}
__device__ __forceinline__ int uneed(int j, int kk) {
    if (j < 7) return kk * HSTEP + 128 + (j + 1) * 8;
    if (j < 63) { const int r = (j - 7) / 14; return U_HEAD + (r * 4 + kk) * RSTEP + (j - 7 - r * 14 + 1) * 8; }
    return U_LAST + (kk + 1) * 8;
}
__device__ __forceinline__ void unit_load_lds(const Params& p, int u, LAS u8* dst, int lane) {
    const UnitDesc d = unit_desc(u);
    const int s = lane >> 2, cqp = lane & 3;
    if (d.isx) {
        const int cq = cqp ^ ((s >> 3) << 1);
        const u8* src = p.x + (size_t)(d.a * 256 + (d.b >> 2) * 128 + (d.b & 3) * 32 + s) * KS + d.kk * 128 + cq * 16;
#pragma unroll
        for (int i = 0; i < 4; ++i) __builtin_amdgcn_global_load_lds((const unsigned*)(src + (size_t)((i >> 1) * 16) * KS + (i & 1) * 64), (LAS unsigned*)(dst + i * 1024), 16, 0, 0);
    } else {
        const int wc = d.b >> 1, kh = d.b & 1;
        const u8* src = p.w + (size_t)(d.kk * 128 + kh * 64 + s) * N + d.a * 256 + wc * 64 + cqp * 16;
#pragma unroll
        for (int i = 0; i < 4; ++i) __builtin_amdgcn_global_load_lds((const unsigned*)(src + (size_t)(i * 16) * N), (LAS unsigned*)(dst + i * 1024), 16, 0, 0);
    }
}
__device__ __forceinline__ void unit_transform(bool isx, const LAS u8* tl, int lane, u32x4 (&cur)[4]) {
    if (isx) {
#pragma unroll
        for (int i = 0; i < 4; ++i) cur[i] = *(const LAS u32x4*)(tl + i * 1024 + lane * 16);
    } else {
        const int s = lane >> 2, cqp = lane & 3;
        const int cq = cqp ^ ((s >> 3) << 1);
        const int rd_off = (cq * 16) * 64 + s * 4;
        unsigned ld[16];
#pragma unroll
        for (int i = 0; i < 16; ++i) ld[i] = *(const LAS unsigned*)(tl + rd_off + i * 64);
#pragma unroll
        for (int t = 0; t < 4; ++t) {
            const unsigned d0 = ld[4 * t], d1 = ld[4 * t + 1], d2 = ld[4 * t + 2], d3 = ld[4 * t + 3];
            const unsigned a = __builtin_amdgcn_perm(d1, d0, 0x05010400u), b = __builtin_amdgcn_perm(d1, d0, 0x07030602u);
            const unsigned c = __builtin_amdgcn_perm(d3, d2, 0x05010400u), e = __builtin_amdgcn_perm(d3, d2, 0x07030602u);
            cur[0][t] = __builtin_amdgcn_perm(c, a, 0x05040100u); cur[1][t] = __builtin_amdgcn_perm(c, a, 0x07060302u);
            cur[2][t] = __builtin_amdgcn_perm(e, b, 0x05040100u); cur[3][t] = __builtin_amdgcn_perm(e, b, 0x07060302u);
        }
    }
}
__device__ __forceinline__ unsigned unit_dst(const UnitDesc& d, int rank, int i, int lane) {
    if (d.isx) return (unsigned)(((d.kk * 8 + rank) * NI + d.a) * BLK + d.b * 4096 + i * 1024 + lane * 16);
    const int wc = d.b >> 1, kh = d.b & 1;
    return (unsigned)(A_BYTES + (size_t)((d.kk * 8 + rank) * NJ + d.a) * BLK + (i >> 1) * HTB + ((wc * 2 + (i & 1)) * 2 + kh) * 1024 + lane * 16);
}

#define RING 2
#define PRE 2
#define CLD 1
constexpr int CL_RING = 0, CL_RAW = RING * 4096, CL_CTL = CL_RAW + (PRE + 1) * 4096;
static_assert(CL_CTL + 256 <= LKNOWN_OFF, "comm LDS");
__device__ __forceinline__ void comm_role(const Params& p, unsigned call, int cwg, int wid, LAS u8* lds) {
    const int lane = threadIdx.x & 63;
    const int rank = p.rank;
    volatile LAS int* ctl = (volatile LAS int*)(lds + CL_CTL);
    if (threadIdx.x < 9) ctl[threadIdx.x] = threadIdx.x == 8 ? 0x7fffffff : 0;
    __syncthreads();
    if (wid == 0) {
        __builtin_amdgcn_s_setprio(3);
        unsigned* claim = (unsigned*)(p.ws[rank] + WS_CLAIM);
        const unsigned cbase = (call - 1u) * (unsigned)(U_TOTAL + C_WG * CLD);
        unsigned long long sexec;
        asm volatile("v_accvgpr_write_b32 a3, 1\n\ts_nop 1" ::: "a3");
#define CLAIM(reg) asm volatile("s_mov_b64 %0, exec\n\ts_mov_b64 exec, 1\n\tglobal_atomic_add a" #reg ", %1, a3, off sc0\n\ts_mov_b64 exec, %0" : "=&s"(sexec) : "v"(claim) : "memory", "a0", "a1", "a2", "a3")
        CLAIM(0); if (CLD > 1) CLAIM(1); if (CLD > 2) CLAIM(2);
        int j = 0, nl = 0x7fffffff;
        int lslot = 0, pslot = 0;
        bool fin = false;
#define STEP(reg) do { \
            if (nl == 0x7fffffff) { \
                if (j < CLD) WAIT_V(0); else asm volatile("s_waitcnt vmcnt(%0)" :: "n"(5 * (CLD - 1)) : "memory"); \
                unsigned cv; asm volatile("v_accvgpr_read_b32 %0, a" #reg : "=v"(cv)); \
                const int id = (int)((unsigned)__builtin_amdgcn_readfirstlane(cv) - cbase); \
                if (id < U_TOTAL) { unit_load_lds(p, id, lds + CL_RAW + lslot * 4096, lane); if (lane == 0) ctl[16 + lslot] = id; CLAIM(reg); if (++lslot > PRE) lslot = 0; } \
                else { nl = j; if (lane == 0) ctl[8] = nl; } \
            } \
            const int k = j - PRE; \
            if (k >= 0) { \
                if (k >= nl) { fin = true; } else { \
                    if (nl == 0x7fffffff) asm volatile("s_waitcnt vmcnt(%0)" :: "n"(5 * PRE + 1) : "memory"); else WAIT_V(0); \
                    const int id = __builtin_amdgcn_readfirstlane(ctl[16 + pslot]); \
                    u32x4 cur[4]; \
                    unit_transform(unit_desc(id).isx, lds + CL_RAW + pslot * 4096, lane, cur); \
                    if (k >= RING) { for (;;) { int mn = ctl[1 + (lane % 7)]; \
                            _Pragma("unroll") for (int o = 4; o; o >>= 1) { const int x = __shfl_xor(mn, o); mn = x < mn ? x : mn; } \
                            if (__builtin_amdgcn_readfirstlane(mn) > k - RING) break; \
                            __builtin_amdgcn_s_sleep(1); } } \
                    LAS u8* rg = lds + CL_RING + (k % RING) * 4096 + lane * 16; \
                    _Pragma("unroll") for (int i = 0; i < 4; ++i) *(LAS u32x4*)(rg + i * 1024) = cur[i]; \
                    if (lane == 0) ctl[32 + (k % RING)] = id; \
                    WAIT_L(0); \
                    if (lane == 0) ctl[0] = k + 1; \
                    if (++pslot > PRE) pslot = 0; \
                } } \
            ++j; } while (0)
        while (!fin) { STEP(0); if (fin) break; if (CLD > 1) { STEP(1); if (fin) break; } if (CLD > 2) STEP(2); }
        WAIT_V(0);
        __builtin_amdgcn_s_setprio(0);
    } else {
        const int dest = (rank + wid) & 7;
        u8* dws = p.ws[dest];
        const __amdgpu_buffer_rsrc_t rs = __builtin_amdgcn_make_buffer_rsrc(dws + WS_DATA + (size_t)(call & 1) * PAR_BYTES, 0, (int)PAR_BYTES, 0x00020000);
        unsigned* prog = (unsigned*)(dws + WS_FLAGS) + rank * 32 + cwg;
        const unsigned pbase = call * PROG_MUL;
        int hist[SWIN];
#pragma unroll
        for (int i = 0; i < SWIN; ++i) hist[i] = -1;
        for (int k = 0;; ++k) {
            bool end = false;
            while (__builtin_amdgcn_readfirstlane(ctl[0]) <= k) { if (__builtin_amdgcn_readfirstlane(ctl[8]) <= k) { end = true; break; } __builtin_amdgcn_s_sleep(1); }
            if (end) break;
            if (__builtin_amdgcn_readfirstlane(ctl[0]) - k >= 3) __builtin_amdgcn_s_setprio(2); else __builtin_amdgcn_s_setprio(0);
            const LAS u8* rg = lds + CL_RING + (k % RING) * 4096 + lane * 16;
            u32x4 cur[4];
#pragma unroll
            for (int i = 0; i < 4; ++i) cur[i] = *(const LAS u32x4*)(rg + i * 1024);
            const int id = __builtin_amdgcn_readfirstlane(ctl[32 + (k % RING)]);
            WAIT_L(0);
            if (lane == 0) ctl[wid] = k + 1;
            const UnitDesc d = unit_desc(id);
#pragma unroll
            for (int i = 0; i < 4; ++i) __builtin_amdgcn_raw_buffer_store_b128(cur[i], rs, unit_dst(d, rank, i, lane), 0, RAUX);
            asm volatile("s_waitcnt vmcnt(%0)" :: "n"(4 * SWIN) : "memory");
            const int oldest = k >= SWIN - 1 ? hist[SWIN - 2 >= 0 ? SWIN - 2 : 0] : (k == 0 ? id : -1);
#pragma unroll
            for (int i = SWIN - 1; i > 0; --i) hist[i] = hist[i - 1];
            hist[0] = id;
            const int X = SWIN == 1 ? id : oldest;
            if (X >= 0 && lane == 0) __hip_atomic_store(prog, pbase + (unsigned)X, __ATOMIC_RELAXED, __HIP_MEMORY_SCOPE_SYSTEM);
        }
        __builtin_amdgcn_s_setprio(0);
        WAIT_V(0);
        if (lane == 0) __hip_atomic_store(prog, pbase + (unsigned)(PROG_MUL - 1), __ATOMIC_RELAXED, __HIP_MEMORY_SCOPE_SYSTEM);
    }
}

#define SA(b, h) (((b) * 2 + (h)) * HTB)
#define SB(b, h) ((4 + (b) * 2 + (h)) * HTB)
#define GLDS(gptr, ldsoff) do { const char* _g = (const char*)(gptr); asm volatile("" : "+s"(_g)); \
        __builtin_amdgcn_global_load_lds((const unsigned*)(_g + voff), (LAS unsigned*)(lds + (ldsoff) + ldsw), 16, 0, 0); } while (0)
#define STAGE(bufoff, gbase) do { GLDS((const char*)(gbase), (bufoff)); GLDS((const char*)(gbase) + 8192, (bufoff) + 8192); } while (0)
#define LDF(off) __builtin_shufflevector(*(const LAS i32x4*)(lds + (off)), *(const LAS i32x4*)(lds + (off) + 1024), 0, 1, 2, 3, 4, 5, 6, 7)
#define LDA(dst, b, h) do { _Pragma("unroll") for (int m = 0; m < 4; ++m) dst[m] = LDF(SA(b, h) + aoff + m * 2048); } while (0)
#define LDB(dst, b, h) do { _Pragma("unroll") for (int n = 0; n < 2; ++n) dst[n] = LDF(SB(b, h) + boff + n * 2048); } while (0)
#define C10(p) "a" #p "0","a" #p "1","a" #p "2","a" #p "3","a" #p "4","a" #p "5","a" #p "6","a" #p "7","a" #p "8","a" #p "9"
#define ACC_CLOB "a0","a1","a2","a3","a4","a5","a6","a7","a8","a9", C10(1),C10(2),C10(3),C10(4),C10(5),C10(6),C10(7),C10(8),C10(9),C10(10),C10(11),"a120","a121","a122","a123","a124","a125","a126","a127"
#define MF(o, A, B) "v_mfma_f32_16x16x128_f8f6f4 a[%c6+" #o ":%c6+" #o "+3], " A ", " B ", a[%c6+" #o ":%c6+" #o "+3]\n\t"
#define MFZ(o, A, B) "v_mfma_f32_16x16x128_f8f6f4 a[%c6+" #o ":%c6+" #o "+3], " A ", " B ", 0\n\t"
#define MMA(ai, bj, At, Bt) asm volatile("s_setprio 1\n\t" MF(0, "%0", "%4") MF(4, "%0", "%5") MF(8, "%1", "%4") MF(12, "%1", "%5") MF(16, "%2", "%4") MF(20, "%2", "%5") MF(24, "%3", "%4") MF(28, "%3", "%5") "s_setprio 0" \
        :: "v"(At[0]), "v"(At[1]), "v"(At[2]), "v"(At[3]), "v"(Bt[0]), "v"(Bt[1]), "i"(((ai) * 2 + (bj)) * 32) : ACC_CLOB)
#define MMAZ(ai, bj, At, Bt) asm volatile("s_setprio 1\n\t" MFZ(0, "%0", "%4") MFZ(4, "%0", "%5") MFZ(8, "%1", "%4") MFZ(12, "%1", "%5") MFZ(16, "%2", "%4") MFZ(20, "%2", "%5") MFZ(24, "%3", "%4") MFZ(28, "%3", "%5") "s_setprio 0" \
        :: "v"(At[0]), "v"(At[1]), "v"(At[2]), "v"(At[3]), "v"(Bt[0]), "v"(Bt[1]), "i"(((ai) * 2 + (bj)) * 32) : ACC_CLOB)
#define MM(Z, ai, bj, At, Bt) do { if (Z) MMAZ(ai, bj, At, Bt); else MMA(ai, bj, At, Bt); } while (0)
#define ACC_RD4(v, i0, i1, i2, i3) asm volatile("v_accvgpr_read_b32 %0, a[%c4]\n\tv_accvgpr_read_b32 %1, a[%c5]\n\tv_accvgpr_read_b32 %2, a[%c6]\n\tv_accvgpr_read_b32 %3, a[%c7]" \
        : "=v"((v)[0]), "=v"((v)[1]), "=v"((v)[2]), "=v"((v)[3]) : "i"(i0), "i"(i1), "i"(i2), "i"(i3))
#define EPI1(ai, m, r) do { float x_[4]; ACC_RD4(x_, (ai) * 64 + (m) * 8 + (r), (ai) * 64 + (m) * 8 + 4 + (r), (ai) * 64 + 32 + (m) * 8 + (r), (ai) * 64 + 32 + (m) * 8 + 4 + (r)); \
        f32x4 v = (f32x4){x_[0], x_[1], x_[2], x_[3]} * scl; \
        v[0] = v[0] < 0.f ? 0.f : v[0]; v[1] = v[1] < 0.f ? 0.f : v[1]; v[2] = v[2] < 0.f ? 0.f : v[2]; v[3] = v[3] < 0.f ? 0.f : v[3]; \
        __builtin_amdgcn_raw_buffer_store_b128(__builtin_bit_cast(u32x4, v), rso, eoff + (unsigned)(((ai) * 128 + (m) * 16 + (r)) * N * 4), 0, 0); } while (0)
#define EPIM(ai, m) do { EPI1(ai, m, 0); EPI1(ai, m, 1); EPI1(ai, m, 2); EPI1(ai, m, 3); } while (0)

__global__ __launch_bounds__(512, 2) void fused_k(Params p) {
    extern __shared__ __attribute__((aligned(16))) u8 lds_raw[];
    LAS u8* lds = (LAS u8*)lds_raw;
    const int tid = threadIdx.x, wid = __builtin_amdgcn_readfirstlane(tid >> 6);
    const int bx = blockIdx.x, rank = p.rank;
    u8* myws = p.ws[rank];
    const unsigned call = (unsigned)__builtin_amdgcn_readfirstlane(__hip_atomic_load((unsigned*)(myws + WS_CALL), __ATOMIC_RELAXED, __HIP_MEMORY_SCOPE_AGENT)) + 1u;
    unsigned* tmo = (unsigned*)(myws + WS_TMO);
    const int wr = wid >> 2, wc = wid & 3;
    const unsigned ldsw = (unsigned)wid * 1024u;
    volatile LAS int* lknown = (volatile LAS int*)(lds + LKNOWN_OFF);
    const size_t par = WS_DATA + (size_t)(call & 1) * PAR_BYTES;
    const char* Abase = (const char*)myws + par;
    const char* Bbase = Abase + A_BYTES;
    const unsigned* flags = (const unsigned*)(myws + WS_FLAGS);
    const float sc = p.sx[0] * p.sw[0];
    int known = 0;
    int knv = 0;
    bool bad = false;
    auto ready_m = [&](int need, unsigned smask) {
        if (known >= need) return;
        { const int ln = threadIdx.x & 63; if (__ballot(ln < 8 && ((smask >> ln) & 1u) && knv < need) == 0ull) return; }
        if (wid == 0) {
            const int lane = threadIdx.x & 63, c = lane & 31;
            const unsigned pbase = call * PROG_MUL;
            const unsigned long long t0 = __builtin_amdgcn_s_memrealtime();
            unsigned pre[4];
            for (;;) {
                unsigned v[4];
#pragma unroll
                for (int i = 0; i < 4; ++i) { const int src = (lane >> 5) + 2 * i;
                    v[i] = src == rank ? __hip_atomic_load((unsigned*)(myws + WS_LOCAL), __ATOMIC_RELAXED, __HIP_MEMORY_SCOPE_AGENT)
                                       : __hip_atomic_load(flags + src * 32 + c, __ATOMIC_RELAXED, __HIP_MEMORY_SCOPE_SYSTEM); }
                unsigned worst = 0x7fffffffu;
#pragma unroll
                for (int i = 0; i < 4; ++i) { const int src = (lane >> 5) + 2 * i;
                    if (src == rank) pre[i] = v[i] >= call * (unsigned)G_WG ? 0x7ffffff0u : 0u;
                    else pre[i] = v[i] >= pbase ? v[i] - pbase : 0u;
#pragma unroll
                    for (int o = 16; o; o >>= 1) { const unsigned x = (unsigned)__shfl_xor((int)pre[i], o); pre[i] = x < pre[i] ? x : pre[i]; }
                    if ((smask >> src) & 1u) worst = pre[i] < worst ? pre[i] : worst; }
                { const unsigned x = (unsigned)__shfl_xor((int)worst, 32); worst = x < worst ? x : worst; }
                const int kn = __builtin_amdgcn_readfirstlane((int)worst);
                if (kn >= need) break;
                if ((unsigned)__builtin_amdgcn_readfirstlane(__hip_atomic_load(tmo, __ATOMIC_RELAXED, __HIP_MEMORY_SCOPE_AGENT)) == call || __builtin_amdgcn_s_memrealtime() - t0 > DEADLINE) {
                    if (lane == 0) __hip_atomic_store(tmo, call, __ATOMIC_RELAXED, __HIP_MEMORY_SCOPE_AGENT);
                    pre[0] = pre[1] = pre[2] = pre[3] = 0x7fffffffu; break; }
                { const unsigned long long w0 = __builtin_amdgcn_s_memrealtime(); const unsigned long long dt = (unsigned long long)((need - kn) * POLL_NUM / 16 + 30);
                  while (__builtin_amdgcn_s_memrealtime() - w0 < dt) __builtin_amdgcn_s_sleep(8); }
            }
            __builtin_amdgcn_fence(__ATOMIC_ACQUIRE, "");
            if (c == 0) {
#pragma unroll
                for (int i = 0; i < 4; ++i) lknown[(lane >> 5) + 2 * i] = (int)pre[i]; }
            asm volatile("s_waitcnt vmcnt(0) lgkmcnt(0)" ::: "memory");
        }
        asm volatile("" ::: "memory"); BAR; asm volatile("" ::: "memory");
        knv = lknown[threadIdx.x & 7];
        asm volatile("s_waitcnt lgkmcnt(0)" ::: "memory");
        { int mn = knv;
#pragma unroll
          for (int o = 4; o; o >>= 1) { const int x = __shfl_xor(mn, o); mn = x < mn ? x : mn; }
          known = __builtin_amdgcn_readfirstlane(mn); }
        if (known == 0x7fffffff) bad = true;
    };
    auto ready = [&](int need) { ready_m(need, 0xffu); };
    auto smask2 = [&](int n) { return (1u << ((rank + n) & 7)) | (1u << ((rank + n + 1) & 7)); };
    auto kt = [&](int n) { return (n & ~7) | ((rank + n) & 7); };
#define APTR(base, n) ((base) + (size_t)kt(n) * (NI * BLK))
#define BPTR(base, n) ((base) + (size_t)kt(n) * (NJ * BLK))
    i32x8 At[4], B0[2], B1[2];
    int sidx = 1; (void)sidx;

    STAMP(0);
    if (is_comm(bx)) {
        comm_role(p, call, bx, wid, lds);
        STAMP(1);
    } else {
        const int lane = tid & 63, fr = lane & 15, fq = lane >> 4;
        const unsigned voff = (unsigned)tid * 16u;
        const int aoff = lds_byte(wr * 64 + fr, fq * 16), boff = lds_byte(wc * 32 + fr, fq * 16);
        const int xcd = bx & 7, loc = (bx >> 3) - C_WG / 8;
        {
            const __amdgpu_buffer_rsrc_t rsl = __builtin_amdgcn_make_buffer_rsrc(myws + par, 0, (int)PAR_BYTES, 0x00020000);
            constexpr int NGW = G_WG * 8;
            static_assert(2 * NGW >= U_TOTAL, "prologue");
            const int gw = (bx - C_WG) * 8 + wid;
            const bool l1 = gw + NGW < U_TOTAL;
            LAS u8* tl = lds + wid * 8192;
            unit_load_lds(p, gw, tl, lane);
            if (l1) unit_load_lds(p, gw + NGW, tl + 4096, lane);
            WAIT_V(0);
            { const UnitDesc d0 = unit_desc(gw); u32x4 c0[4]; unit_transform(d0.isx, tl, lane, c0);
#pragma unroll
              for (int i = 0; i < 4; ++i) __builtin_amdgcn_raw_buffer_store_b128(c0[i], rsl, unit_dst(d0, rank, i, lane), 0, 16); }
            if (l1) { const UnitDesc d1 = unit_desc(gw + NGW); u32x4 c1[4]; unit_transform(d1.isx, tl + 4096, lane, c1);
#pragma unroll
                for (int i = 0; i < 4; ++i) __builtin_amdgcn_raw_buffer_store_b128(c1[i], rsl, unit_dst(d1, rank, i, lane), 0, 16); }
            asm volatile("s_waitcnt vmcnt(0) lgkmcnt(0)" ::: "memory");
            __syncthreads();
            if (tid == 0) __hip_atomic_fetch_add((unsigned*)(myws + WS_LOCAL), 1u, __ATOMIC_RELAXED, __HIP_MEMORY_SCOPE_AGENT);
            STAMP(30);
        }
    }
    {
        asm volatile("s_waitcnt vmcnt(0) lgkmcnt(0)" ::: "memory");
        BAR;
        int tz = threadIdx.x; asm volatile("" : "+v"(tz));
        const int lane = tz & 63, fr = lane & 15, fq = lane >> 4;
        const unsigned voff = (unsigned)tz * 16u;
        const int aoff = lds_byte(wr * 64 + fr, fq * 16), boff = lds_byte(wc * 32 + fr, fq * 16);
        int ti, tj, ah;
        if (is_comm(bx)) { ti = bx >> 1; ah = bx & 1; tj = 63; }
        else { const int loc = (bx >> 3) - C_WG / 8; ti = 4 * (bx & 3) + (loc & 3); tj = loc >> 2; ah = (bx >> 2) & 1; }
        const int gate = is_comm(bx) ? 63 : 6;
        const char* cA = Abase + (size_t)ti * BLK + ah * HTB;
        const char* cB = Bbase + (size_t)tj * BLK;
        ready_m(uneed(gate, 0), smask2(0));
        STAMP(sidx); ++sidx;
        constexpr int HB = 49152;
        { const char* a = APTR(cA, 0); const char* b = BPTR(cB, 0); STAGE(0, a); STAGE(HTB, b); STAGE(2 * HTB, b + HTB);
          a = APTR(cA, 1); b = BPTR(cB, 1); STAGE(HB, a); STAGE(HB + HTB, b); STAGE(HB + 2 * HTB, b + HTB); }
        if (wr == 1) BAR;
        WAIT_V(6); BAR; BAR;
        int bo = 0;
        for (int t = 0; t < NT; ++t) {
            int nbo = bo + 2 * HB; if (nbo >= 3 * HB) nbo -= 3 * HB;
            const bool more = t + 2 < NT;
            if (more) ready_m(uneed(gate, (t + 2) >> 3), 1u << ((rank + t + 2) & 7));
            const char* a = APTR(cA, (more ? t + 2 : t)); const char* b = BPTR(cB, (more ? t + 2 : t));
#pragma unroll
            for (int n = 0; n < 2; ++n) B0[n] = LDF(bo + HTB + boff + n * 2048);
#pragma unroll
            for (int m = 0; m < 4; ++m) At[m] = LDF(bo + aoff + m * 2048);
#pragma unroll
            for (int n = 0; n < 2; ++n) B1[n] = LDF(bo + 2 * HTB + boff + n * 2048);
            if (more) { STAGE(nbo, a); STAGE(nbo + HTB, b); STAGE(nbo + 2 * HTB, b + HTB); WAIT_V(6); } else WAIT_V(0);
            WAIT_L(0); BAR; MM(t == 0, 0, 0, At, B0); MM(t == 0, 0, 1, At, B1); BAR; SCHED;
            bo += HB; if (bo >= 3 * HB) bo = 0;
        }
        if (wr == 0) BAR;
        asm volatile("s_nop 15\n\ts_nop 15" ::: "memory");
        const float scl = bad ? __builtin_nanf("") : sc;
        const __amdgpu_buffer_rsrc_t rso = __builtin_amdgcn_make_buffer_rsrc(p.out, 0, M * N * 4, 0x00020000);
        const unsigned eoff = (unsigned)(((ti * 256 + ah * 128 + wr * 64 + 4 * fq) * N + tj * 256 + wc * 64 + 4 * fr) * 4);
        EPIM(0, 0); EPIM(0, 1); EPIM(0, 2); EPIM(0, 3);
        STAMP(sidx); ++sidx;
    }
    if (!is_comm(bx)) {
        int tz = threadIdx.x; asm volatile("" : "+v"(tz));
        const int lane = tz & 63, fr = lane & 15, fq = lane >> 4;
        const unsigned voff = (unsigned)tz * 16u;
        const int aoff = lds_byte(wr * 64 + fr, fq * 16), boff = lds_byte(wc * 32 + fr, fq * 16);
        const int xcd = bx & 7, loc = (bx >> 3) - C_WG / 8;
        const int ti = 4 * (xcd & 3) + (loc & 3);
        int tj = 7 + 7 * (xcd >> 2) + (loc >> 2), gate = 7 + 7 * (xcd >> 2) + 6;
        const char* cA = Abase + (size_t)ti * BLK;
        const char* cB = Bbase + (size_t)tj * BLK;
        ready_m(uneed(gate, 0), smask2(0));
        STAMP(sidx); ++sidx;
        {
            const char* a0 = APTR(cA, 0); const char* b0 = BPTR(cB, 0); const char* a1 = APTR(cA, 1); const char* b1 = BPTR(cB, 1);
            STAGE(SB(0, 0), b0); STAGE(SB(0, 1), b0 + HTB); STAGE(SA(0, 0), a0); STAGE(SA(0, 1), a0 + HTB);
            STAGE(SB(1, 0), b1); STAGE(SB(1, 1), b1 + HTB); STAGE(SA(1, 0), a1);
            if (wr == 1) BAR;
            WAIT_V(6); BAR; BAR;
        }
        bool first = true; (void)first;
#define KT(b, MODE) do { \
                LDB(B0, b, 0); LDB(B1, b, 1); SCHED; LDA(At, b, 0); STAGE(SA((b) ^ 1, 1), an + HTB); \
                if (MODE == 1) { SCHED; EPIM(1, 0); EPIM(1, 1); EPIM(1, 2); EPIM(1, 3); SCHED; WAIT_V(40); } else if (MODE == 2) WAIT_V(24); else WAIT_V(8); \
                WAIT_L(0); BAR; MM(MODE == 1, 0, 0, At, B0); MM(MODE == 1, 0, 1, At, B1); BAR; SCHED; \
                LDA(At, b, 1); STAGE(SB(b, 0), b2); STAGE(SB(b, 1), b2 + HTB); STAGE(SA(b, 0), a2); \
                if (MODE == 3) { SCHED; asm volatile("s_nop 7"); EPIM(0, 0); EPIM(0, 1); EPIM(0, 2); EPIM(0, 3); SCHED; WAIT_V(24); } else if (MODE == 1) WAIT_V(40); else WAIT_V(8); \
                WAIT_L(0); BAR; MM(MODE == 1, 1, 0, At, B0); MM(MODE == 1, 1, 1, At, B1); BAR; SCHED; } while (0)
        const unsigned eoff0 = (unsigned)(((ti * 256 + wr * 64 + 4 * fq) * N + wc * 64 + 4 * fr) * 4);
        __amdgpu_buffer_rsrc_t rso = __builtin_amdgcn_make_buffer_rsrc(p.out, 0, 0, 0x00020000);
        unsigned eoff = eoff0;
        float scl = sc;
        for (;;) {
            const bool has_next = tj + 14 < 63;
            const int ntj = tj + 14, ngate = gate + 14;
            const char* nB = has_next ? Bbase + (size_t)ntj * BLK : cB;
            {
                ready_m(uneed(gate, 0), smask2(2));
                asm volatile("s_nop 7");
                { const char* an = APTR(cA, 1); const char* a2 = APTR(cA, 2); const char* b2 = BPTR(cB, 2); KT(0, 1); }
                { const char* an = APTR(cA, 2); const char* a2 = APTR(cA, 3); const char* b2 = BPTR(cB, 3); KT(1, 2); }
            }
            for (int t = 2; t < NT - 2; t += 2) {
                ready_m(uneed(gate, (t + 2) >> 3), smask2(t + 2));
                { const char* an = APTR(cA, t + 1); const char* a2 = APTR(cA, t + 2); const char* b2 = BPTR(cB, t + 2); KT(0, 0); }
                { const char* an = APTR(cA, t + 2); const char* a2 = APTR(cA, t + 3); const char* b2 = BPTR(cB, t + 3); KT(1, 0); }
            }
            {
                if (has_next) ready_m(uneed(ngate, 0), smask2(0));
                { const char* an = APTR(cA, NT - 1); const char* a2 = APTR(cA, 0); const char* b2 = BPTR(nB, 0); KT(0, 0); }
                rso = __builtin_amdgcn_make_buffer_rsrc(p.out, 0, M * N * 4, 0x00020000);
                eoff = eoff0 + (unsigned)tj * 1024u;
                scl = bad ? __builtin_nanf("") : sc;
                { const char* an = APTR(cA, 0); const char* a2 = APTR(cA, 1); const char* b2 = BPTR(nB, 1); KT(1, 3); }
            }
            STAMP(sidx); ++sidx;
            if (!has_next) break;
            gate = ngate; tj = ntj; cB = nB; first = false;
        }
        asm volatile("s_nop 15\n\ts_nop 15" ::: "memory");
        EPIM(1, 0); EPIM(1, 1); EPIM(1, 2); EPIM(1, 3);
        WAIT_V(0);
        if (wr == 0) BAR;
    }
    __syncthreads();
    if (tid == 0) {
        const unsigned t = __hip_atomic_fetch_add((unsigned*)(myws + WS_DONE), 1u, __ATOMIC_RELAXED, __HIP_MEMORY_SCOPE_AGENT);
        if (t == call * GRID - 1u) __hip_atomic_store((unsigned*)(myws + WS_CALL), call, __ATOMIC_RELAXED, __HIP_MEMORY_SCOPE_AGENT);
    }
}
}

extern "C" void kernel_launch(int rank, int nranks, ncclComm_t comm, hipStream_t stream,
                              void* const* d_in, const int* in_sizes, int n_in,
                              void* d_out, int out_size,
                              const int* in_dtypes, int out_dtype,
                              void* const* d_out_all,
                              void* const* d_ws_all, size_t ws_bytes) {
    if (nranks != NR || rank < 0 || rank >= NR || ws_bytes < WS_NEED || n_in < 4 || out_size != M * N) { fprintf(stderr, "unexpected configuration\n"); return; }
    HIP_CHECK(hipFuncSetAttribute((const void*)fused_k, hipFuncAttributeMaxDynamicSharedMemorySize, LDS_BYTES));
    Params p;
    p.x = (const u8*)d_in[0]; p.w = (const u8*)d_in[1]; p.sx = (const float*)d_in[2]; p.sw = (const float*)d_in[3];
    p.out = (float*)d_out; p.rank = rank;
    for (int i = 0; i < NR; ++i) p.ws[i] = (u8*)d_ws_all[i];
    hipLaunchKernelGGL(fused_k, dim3(GRID), dim3(512), LDS_BYTES, stream, p);
    HIP_CHECK(hipGetLastError());
}
```
